```python
import numpy as np

kernel_name = "gpu_gemm_a2a_m4096_k4096_n4096_bf16_q8"


ABSMAX_RTOL = 0.02

COMPUTE_DTYPE = "bfloat16"
EPILOGUE = "quant_int8"
IN_QUANT = None
OUT_QUANT = "int8"
IN_DTYPES = {"x": "bfloat16", "w_mat": "bfloat16"}
WORLD_SIZE = 8

def _to_bf16(x):
    u = np.ascontiguousarray(x, dtype=np.float32).view(np.uint32)
    return ((u + 0x7FFF + ((u >> 16) & 1)) & 0xFFFF0000).view(np.float32)

def setup_inputs():

    x = _to_bf16(np.random.randn(4096, 4096).astype(np.float32))
    w_mat = _to_bf16((np.random.randn(4096, 4096) / np.sqrt(4096)).astype(np.float32))
    return {"x": x, "w_mat": w_mat}

def _quant_dequant(y, amax):
    scale = amax / 127.0
    q = np.clip(np.round(y / scale), -127.0, 127.0)
    return (q * scale).astype(np.float32)
def _epilogue(y):
    return _quant_dequant(y, np.abs(y).max())

def reference(x, w_mat):
    y = (x @ w_mat).astype(np.float32)
    return _epilogue(y)
```

```cpp
#include <hip/hip_runtime.h>
#include <rccl/rccl.h>
#include <cstdio>

typedef unsigned short u16;
typedef unsigned int u32;
typedef unsigned long long u64;
typedef __attribute__((ext_vector_type(8))) short s16x8;
typedef __attribute__((ext_vector_type(4))) short s16x4;
typedef __attribute__((ext_vector_type(8))) __bf16 bf16x8;
typedef __attribute__((ext_vector_type(4))) float f32x4;
typedef __attribute__((ext_vector_type(4))) unsigned int u32x4;
#define LDS __attribute__((address_space(3)))
#define GAS __attribute__((address_space(1)))

namespace {
constexpr int NR = 8;
constexpr int MS = 512, N = 4096, K = 4096;
constexpr int NSH = N / NR;
constexpr int BM = 128, BN = 128, BK = 64, KSPLIT = 2, NS = 4;
constexpr int WBYTES = BK * BN * 2, XBYTES = BM * BK * 2, STAGE = WBYTES + XBYTES;
constexpr int T = K / KSPLIT / BK;
constexpr int NWG = 256;
constexpr int LDS_BYTES = NS * STAGE;
constexpr int LPW = 8;
constexpr size_t WS_CALL = 0;
constexpr size_t WS_TMO = 128;
constexpr size_t WS_XF = 4096;
constexpr size_t WS_L1G = WS_XF + 256 * 4 * 64;
constexpr int REP = 8;
constexpr size_t WS_GG = WS_L1G + 2 * 8 * 32 * 8;
constexpr size_t WS_DF = WS_GG + 2 * REP * 64 * 8;
constexpr size_t WS_DBG = 512 * 1024;
constexpr size_t WS_SLAB = 1 << 20;
constexpr size_t WS_NEED = WS_SLAB + (size_t)NWG * 32768;
constexpr u64 DEADLINE = 100000000ull;

struct Params {
  const u16* x; const u16* w;
  char* out[NR]; char* ws[NR];
  int rank;
};

template <int N_> __device__ __forceinline__ void wait_vm() { asm volatile("s_waitcnt vmcnt(%0)" ::"i"(N_) : "memory"); }
__device__ __forceinline__ void bar() { asm volatile("s_barrier" ::: "memory"); }
__device__ __forceinline__ void bar_lgkm() {
  __builtin_amdgcn_sched_barrier(0);
  __builtin_amdgcn_s_waitcnt(0xC07F);
  asm volatile("s_barrier" ::: "memory");
  __builtin_amdgcn_sched_barrier(0);
}
__device__ __forceinline__ u32 ld_agent32(const char* p) { return __hip_atomic_load((const GAS u32*)p, __ATOMIC_RELAXED, __HIP_MEMORY_SCOPE_AGENT); }
__device__ __forceinline__ u32 ld_sys32(const char* p) { return __hip_atomic_load((const GAS u32*)p, __ATOMIC_RELAXED, __HIP_MEMORY_SCOPE_SYSTEM); }
__device__ __forceinline__ u64 ld_agent64(const char* p) { return __hip_atomic_load((const GAS u64*)p, __ATOMIC_RELAXED, __HIP_MEMORY_SCOPE_AGENT); }
__device__ __forceinline__ u64 ld_sys64(const char* p) { return __hip_atomic_load((const GAS u64*)p, __ATOMIC_RELAXED, __HIP_MEMORY_SCOPE_SYSTEM); }
__device__ __forceinline__ void st_agent32(char* p, u32 v) { __hip_atomic_store((GAS u32*)p, v, __ATOMIC_RELAXED, __HIP_MEMORY_SCOPE_AGENT); }
__device__ __forceinline__ void st_sys32(char* p, u32 v) { __hip_atomic_store((GAS u32*)p, v, __ATOMIC_RELAXED, __HIP_MEMORY_SCOPE_SYSTEM); }
__device__ __forceinline__ void st_agent64(char* p, u64 v) { __hip_atomic_store((GAS u64*)p, v, __ATOMIC_RELAXED, __HIP_MEMORY_SCOPE_AGENT); }
__device__ __forceinline__ void st_sys64(char* p, u64 v) { __hip_atomic_store((GAS u64*)p, v, __ATOMIC_RELAXED, __HIP_MEMORY_SCOPE_SYSTEM); }

__device__ __forceinline__ bool give_up(char* myws, u32 call, u64 t0) {
  bool bad = ld_agent32(myws + WS_TMO) == call || (__builtin_amdgcn_s_memrealtime() - t0) > DEADLINE;
  bad = __builtin_amdgcn_readfirstlane(bad);
  if (bad) st_agent32(myws + WS_TMO, call);
  return bad;
}
__device__ __forceinline__ float wave_max(float v) {
#pragma unroll
  for (int o = 32; o > 0; o >>= 1) v = fmaxf(v, __shfl_xor(v, o, 64));
  return v;
}
#define STAMP(i) do {} while (0)
}

__global__ __launch_bounds__(512, 2) void fused_gemm_a2a_q8(Params P) {
#if __HIP_DEVICE_COMPILE__
  extern __shared__ __attribute__((aligned(16))) char smem[];
  const int tid = threadIdx.x, lane = tid & 63;
  const int wid = __builtin_amdgcn_readfirstlane(tid >> 6);
  const int b = blockIdx.x, grp = b & 7, idx = b >> 3;
  const int tn = grp * 4 + (idx & 3), tm = (idx >> 2) & 3, kh = idx >> 4;
  const int n0 = tn * BN, m0 = tm * BM, k0 = kh * (K / KSPLIT);
  const int partner = b ^ 128;
  char* const myws = P.ws[P.rank];
  STAMP(0);
  f32x4 acc[4][4];
  u32 call;
  if (wid >= 4) {
    const int lw = wid - 4;
    auto rw = __builtin_amdgcn_make_buffer_rsrc((void*)P.w, 0, K * N * 2, 0x00020000);
    auto rx = __builtin_amdgcn_make_buffer_rsrc((void*)P.x, 0, MS * K * 2, 0x00020000);
    int vw[4], vx[4];
#pragma unroll
    for (int j = 0; j < 4; ++j) {
      int k = 16 * lw + 4 * j + (lane >> 4);
      int F = (k & 3) | ((k >> 1) & 4);
      vw[j] = k * (N * 2) + (((lane & 15) ^ (F << 1)) * 16) + n0 * 2;
      int r = (lw * 4 + j) * 8 + (lane >> 3);
      vx[j] = (m0 + r) * (K * 2) + (((lane & 7) ^ ((r >> 1) & 7)) * 16);
    }
    auto issue = [&](int s, int slot_off) {
      int sw = (s < T) ? (k0 + s * BK) * (N * 2) : 0x40000000;
      int sx = (s < T) ? (k0 + s * BK) * 2 : 0x40000000;
#pragma unroll
      for (int j = 0; j < 4; ++j) __builtin_amdgcn_raw_ptr_buffer_load_lds(rw, (LDS void*)(smem + slot_off + (lw * 4 + j) * 1024), 16, vw[j], sw, 0, 0);
#pragma unroll
      for (int j = 0; j < 4; ++j) __builtin_amdgcn_raw_ptr_buffer_load_lds(rx, (LDS void*)(smem + slot_off + WBYTES + (lw * 4 + j) * 1024), 16, vx[j], sx, 0, 0);
    };
#pragma unroll
    for (int s = 0; s < NS - 1; ++s) issue(s, s * STAGE);
    int slot = (NS - 1) * STAGE;
    for (int s = 0; s < T; ++s) {
      wait_vm<(NS - 2) * LPW>();
      bar();
      issue(s + NS - 1, slot);
      slot += STAGE; if (slot == NS * STAGE) slot = 0;
    }
    wait_vm<0>();
    call = __builtin_amdgcn_readfirstlane(ld_agent32(myws + WS_CALL)) + 1;
    bar();
  } else {
    u32 callv = ld_agent32(myws + WS_CALL);
    const int wm = wid & 1, wn = wid >> 1;
    const int g = lane >> 4, q = (lane & 15) >> 2, p = lane & 3;
    const int F = q | ((g & 1) << 2);
    int aw[4], ax[2];
#pragma unroll
    for (int c = 0; c < 4; ++c) aw[c] = (8 * g + q) * 256 + ((((4 * wn + c) ^ F) << 1) | (p >> 1)) * 16 + 8 * (p & 1);
#pragma unroll
    for (int ks = 0; ks < 2; ++ks) { int r = wm * 64 + (lane & 15); ax[ks] = WBYTES + r * 128 + (((4 * ks + g) ^ ((r >> 1) & 7)) * 16); }
#pragma unroll
    for (int i = 0; i < 4; ++i)
#pragma unroll
      for (int j = 0; j < 4; ++j) acc[i][j] = f32x4{0, 0, 0, 0};
    s16x8 W0[4][2], X0[4][2], W1[4][2], X1[4][2];
    auto rd = [&](s16x8(&W)[4][2], s16x8(&X)[4][2], int so) {
#pragma unroll
      for (int ks = 0; ks < 2; ++ks) {
#pragma unroll
        for (int j = 0; j < 4; ++j) X[j][ks] = *(LDS s16x8*)(smem + so + ax[ks] + j * 16 * 128);
#pragma unroll
        for (int c = 0; c < 4; ++c) {
          s16x4 lo = __builtin_amdgcn_ds_read_tr16_b64_v4i16((LDS s16x4*)(smem + so + aw[c] + (32 * ks) * 256));
          s16x4 hi = __builtin_amdgcn_ds_read_tr16_b64_v4i16((LDS s16x4*)(smem + so + aw[c] + (32 * ks + 4) * 256));
          W[c][ks] = __builtin_shufflevector(lo, hi, 0, 1, 2, 3, 4, 5, 6, 7);
        }
      }
    };
    auto mm = [&](s16x8(&W)[4][2], s16x8(&X)[4][2]) {
#pragma unroll
      for (int ks = 0; ks < 2; ++ks)
#pragma unroll
        for (int c = 0; c < 4; ++c)
#pragma unroll
          for (int j = 0; j < 4; ++j)
            acc[c][j] = __builtin_amdgcn_mfma_f32_16x16x32_bf16(__builtin_bit_cast(bf16x8, W[c][ks]), __builtin_bit_cast(bf16x8, X[j][ks]), acc[c][j], 0, 0, 0);
    };
    auto sched = [&]() {
#pragma unroll
      for (int i = 0; i < 32; ++i) {
        __builtin_amdgcn_sched_group_barrier(0x8, 1, 0);
        if (i < 24) __builtin_amdgcn_sched_group_barrier(0x100, 1, 0);
      }
    };
    int slot = 0;
    bar();
    rd(W0, X0, slot); slot += STAGE;
    for (int s = 1; s < T - 1; s += 2) {
      bar_lgkm();
      rd(W1, X1, slot); mm(W0, X0); sched();
      slot += STAGE; if (slot == NS * STAGE) slot = 0;
      bar_lgkm();
      rd(W0, X0, slot); mm(W1, X1); sched();
      slot += STAGE; if (slot == NS * STAGE) slot = 0;
    }
    bar_lgkm();
    rd(W1, X1, slot); mm(W0, X0); sched();
    bar_lgkm();
    mm(W1, X1);
    call = __builtin_amdgcn_readfirstlane(callv) + 1;
    STAMP(1);
    {
      auto rs = __builtin_amdgcn_make_buffer_rsrc((void*)(myws + WS_SLAB), 0, NWG * 32768, 0x00020000);
      const int so_out = (partner * 4 + wid) * 8192, so_in = (b * 4 + wid) * 8192;
      if (kh == 0) {
#pragma unroll
        for (int c = 0; c < 4; ++c)
#pragma unroll
          for (int j = 0; j < 2; ++j) __builtin_amdgcn_raw_buffer_store_b128(__builtin_bit_cast(u32x4, acc[c][2 + j]), rs, lane * 16 + (c * 2 + j) * 1024, so_out, 16);
      } else {
#pragma unroll
        for (int c = 0; c < 4; ++c)
#pragma unroll
          for (int j = 0; j < 2; ++j) {
            __builtin_amdgcn_raw_buffer_store_b128(__builtin_bit_cast(u32x4, acc[c][j]), rs, lane * 16 + (c * 2 + j) * 1024, so_out, 16);
            acc[c][j] = acc[c][2 + j];
          }
      }
      wait_vm<0>();
      if (lane == 0) st_agent32(myws + WS_XF + (partner * 4 + wid) * 64, call);
      STAMP(2);
      u64 t0 = __builtin_amdgcn_s_memrealtime();
      const char* f = myws + WS_XF + (b * 4 + wid) * 64;
      while (__builtin_amdgcn_readfirstlane(ld_agent32(f)) < call) {
        if (give_up(myws, call, t0)) break;
        __builtin_amdgcn_s_sleep(1);
      }
      __builtin_amdgcn_fence(__ATOMIC_ACQUIRE, "wavefront");
      STAMP(3);
#pragma unroll
      for (int c = 0; c < 4; ++c)
#pragma unroll
        for (int j = 0; j < 2; ++j) {
          u32x4 v = __builtin_amdgcn_raw_buffer_load_b128(rs, lane * 16 + (c * 2 + j) * 1024, so_in, 16);
          acc[c][j] += __builtin_bit_cast(f32x4, v);
        }
    }
    float am = 0.f;
#pragma unroll
    for (int c = 0; c < 4; ++c)
#pragma unroll
      for (int j = 0; j < 2; ++j) {
#pragma unroll
        for (int r = 0; r < 4; ++r) am = fmaxf(am, fabsf(acc[c][j][r]));
        int row = wm * 32 + j * 16 + (lane & 15), col = wn * 64 + c * 16 + 4 * g;
        *(LDS f32x4*)(smem + 1024 + row * 528 + col * 4) = acc[c][j];
      }
    am = wave_max(am);
    if (lane == 0) *(LDS float*)(smem + wid * 4) = am;
    STAMP(4);
  }
  bar_lgkm();
  const int par = call & 1;
  if (wid == 4) {
    f32x4 a4 = *(LDS f32x4*)(smem);
    float am = fmaxf(fmaxf(a4[0], a4[1]), fmaxf(a4[2], a4[3]));
    char* l1 = myws + WS_L1G + (size_t)(par * 8 + grp) * 256;
    if (lane == 0) st_agent64(l1 + idx * 8, ((u64)call << 32) | __float_as_uint(am));
    u64 t0 = __builtin_amdgcn_s_memrealtime();
    bool ok = true;
    if (idx == 0) {
      u64 v;
      for (;;) {
        v = ld_agent64(l1 + (lane & 31) * 8);
        if (__all((u32)(v >> 32) == call)) break;
        if (give_up(myws, call, t0)) { ok = false; break; }
        __builtin_amdgcn_s_sleep(1);
      }
      float gm = wave_max(__uint_as_float((u32)v));
      char* dst = P.ws[lane & 7] + WS_GG + (size_t)((par * REP + (lane >> 3)) * 64 + P.rank * 8 + grp) * 8;
      if (ok) st_sys64(dst, ((u64)call << 32) | __float_as_uint(gm));
    }
    STAMP(5);
    u64 v;
    const char* gsrc = myws + WS_GG + (size_t)((par * REP + (idx & (REP - 1))) * 64 + lane) * 8;
    for (;;) {
      v = ld_sys64(gsrc);
      if (__all((u32)(v >> 32) == call)) break;
      if (give_up(myws, call, t0)) { ok = false; break; }
      __builtin_amdgcn_s_sleep(1);
    }
    float gm = wave_max(__uint_as_float((u32)v));
    if (ld_agent32(myws + WS_TMO) == call) ok = false;
    if (lane == 0) {
      *(LDS float*)(smem + 32) = gm > 0.f ? 127.0f / gm : 0.f;
      *(LDS u32*)(smem + 36) = ok ? 1u : 0u;
      if (b == 0) st_agent32(myws + WS_CALL, call);
    }
    STAMP(6);
  }
  const int row = tid >> 3, cq = tid & 7;
  f32x4 yv[4];
#pragma unroll
  for (int i = 0; i < 4; ++i) yv[i] = *(LDS f32x4*)(smem + 1024 + row * 528 + cq * 64 + i * 16);
  bar_lgkm();
  {
    const float inv = *(LDS float*)(smem + 32);
    const u32 ok = *(LDS u32*)(smem + 36);
    u32x4 o;
#pragma unroll
    for (int i = 0; i < 4; ++i) {
      u32 pk = 0;
#pragma unroll
      for (int r = 0; r < 4; ++r) {
        float qf = __builtin_amdgcn_fmed3f(__builtin_rintf(yv[i][r] * inv), -127.f, 127.f);
        pk |= ((u32)(int)qf & 0xffu) << (8 * r);
      }
      o[i] = ok ? pk : 0x80808080u;
    }
    const int grow = P.rank * MS + m0 + (row >> 5) * 64 + kh * 32 + (row & 31);
    auto ro = __builtin_amdgcn_make_buffer_rsrc((void*)P.out[grp], 0, N * NSH, 0x00020000);
    __builtin_amdgcn_raw_buffer_store_b128(o, ro, grow * NSH + (idx & 3) * BN + cq * 16, 0, 17);
  }
  wait_vm<0>();
  bar();
  if (wid == 4) {
    STAMP(7);
    if (lane == 0 && grp != P.rank) st_sys32(P.ws[grp] + WS_DF + (size_t)(P.rank * 32 + idx) * 4, call);
    if (b == P.rank) {
      u64 t0 = __builtin_amdgcn_s_memrealtime();
      const char* f = myws + WS_DF + lane * 16;
      const bool mine = (lane >> 3) == P.rank;
      for (;;) {
        u32 a0 = ld_sys32(f), a1 = ld_sys32(f + 4), a2 = ld_sys32(f + 8), a3 = ld_sys32(f + 12);
        bool okk = mine || (a0 >= call && a1 >= call && a2 >= call && a3 >= call);
        if (__all(okk)) break;
        if (give_up(myws, call, t0)) {
          if (lane == 0) *(GAS char*)(P.out[P.rank] + (size_t)(P.rank * MS + m0 + kh * 32) * NSH + (idx & 3) * BN) = (char)0x80;
          break;
        }
        __builtin_amdgcn_s_sleep(1);
      }
      STAMP(8);
    }
  }
#endif
}

extern "C" void kernel_launch(int rank, int nranks, ncclComm_t comm, hipStream_t stream, void* const* d_in, const int* in_sizes, int n_in,
                              void* d_out, int out_size, const int* in_dtypes, int out_dtype, void* const* d_out_all,
                              void* const* d_ws_all, size_t ws_bytes) {
  if (nranks != NR || (unsigned)rank >= (unsigned)NR || ws_bytes < WS_NEED || n_in < 2 || in_sizes[0] != MS * K || in_sizes[1] != K * N || out_size != N * NSH) {
    fprintf(stderr, "unexpected problem shape\n");
    return;
  }
  Params P;
  P.x = (const u16*)d_in[0];
  P.w = (const u16*)d_in[1];
  for (int p = 0; p < NR; ++p) { P.out[p] = (char*)d_out_all[p]; P.ws[p] = (char*)d_ws_all[p]; }
  P.rank = rank;
  hipError_t e = hipFuncSetAttribute((const void*)fused_gemm_a2a_q8, hipFuncAttributeMaxDynamicSharedMemorySize, LDS_BYTES);
  if (e != hipSuccess) { fprintf(stderr, "hipFuncSetAttribute: %s\n", hipGetErrorString(e)); return; }
  hipLaunchKernelGGL(fused_gemm_a2a_q8, dim3(NWG), dim3(512), LDS_BYTES, stream, P);
}
```

```cpp
#include <hip/hip_runtime.h>
#include <rccl/rccl.h>
#include <cstdio>

typedef unsigned short u16;
typedef unsigned int u32;
typedef unsigned long long u64;
typedef __attribute__((ext_vector_type(8))) short s16x8;
typedef __attribute__((ext_vector_type(4))) short s16x4;
typedef __attribute__((ext_vector_type(8))) __bf16 bf16x8;
typedef __attribute__((ext_vector_type(2))) __bf16 bf16x2;
typedef __attribute__((ext_vector_type(4))) float f32x4;
typedef __attribute__((ext_vector_type(2))) float f32x2;
typedef __attribute__((ext_vector_type(4))) unsigned int u32x4;
#define LDS __attribute__((address_space(3)))
#define GAS __attribute__((address_space(1)))

#define EARLY 1
#define XMODE 0
#define NSTAGE 4
#define FIN_SLEEP 2
#define WB 0
#define PACE 160
#define PRE_SLEEP 4
#define EARLY_DEPTH 1

namespace {
constexpr int NR = 8;
constexpr int MS = 512, N = 4096, K = 4096;
constexpr int NSH = N / NR;
constexpr int BM = 128, BN = 128, BK = 64, KSPLIT = 2, NS = NSTAGE;
constexpr int WBYTES = BK * BN * 2, XBYTES = BM * BK * 2, STAGE = WBYTES + XBYTES;
constexpr int T = K / KSPLIT / BK;
constexpr int NWG = 256;
constexpr int LDS_BYTES = NS * STAGE;
constexpr int LPW = 8;
constexpr size_t WS_CALL = 0;
constexpr size_t WS_TMO = 128;
constexpr size_t WS_L1G = 4096;
constexpr int REP = 8;
constexpr size_t WS_GG = WS_L1G + 2 * 8 * 128 * 8;
constexpr size_t WS_XID = WS_GG + 2 * REP * 64 * 8;
constexpr size_t WS_DBG = 512 * 1024;
constexpr size_t WS_SLAB = 1 << 20;
constexpr size_t WS_STG = WS_SLAB + (size_t)NWG * 32768;
constexpr size_t STG_BYTES = (size_t)N * NSH * 2;
constexpr size_t WS_NEED = WS_STG + 2 * STG_BYTES;
constexpr u64 DEADLINE = 100000000ull;
constexpr int AUX_SC1 = 16, AUX_SYS = 17;

struct Params {
  const u16* x; const u16* w;
  char* myws; char* myout;
  char* out[NR]; char* ws[NR];
  int rank;
};

template <int N_> __device__ __forceinline__ void wait_vm() { asm volatile("s_waitcnt vmcnt(%0)" ::"i"(N_) : "memory"); }
__device__ __forceinline__ void bar() { asm volatile("s_barrier" ::: "memory"); }
__device__ __forceinline__ void bar_lgkm() {
  __builtin_amdgcn_sched_barrier(0);
  __builtin_amdgcn_s_waitcnt(0xC07F);
  asm volatile("s_barrier" ::: "memory");
  __builtin_amdgcn_sched_barrier(0);
}
__device__ __forceinline__ u32 ld_agent32(const char* p) { return __hip_atomic_load((const GAS u32*)p, __ATOMIC_RELAXED, __HIP_MEMORY_SCOPE_AGENT); }
__device__ __forceinline__ u64 ld_sys64(const char* p) { return __hip_atomic_load((const GAS u64*)p, __ATOMIC_RELAXED, __HIP_MEMORY_SCOPE_SYSTEM); }
__device__ __forceinline__ void st_agent32(char* p, u32 v) { __hip_atomic_store((GAS u32*)p, v, __ATOMIC_RELAXED, __HIP_MEMORY_SCOPE_AGENT); }
__device__ __forceinline__ void st_agent64(char* p, u64 v) { __hip_atomic_store((GAS u64*)p, v, __ATOMIC_RELAXED, __HIP_MEMORY_SCOPE_AGENT); }
__device__ __forceinline__ void st_sys64(char* p, u64 v) { __hip_atomic_store((GAS u64*)p, v, __ATOMIC_RELAXED, __HIP_MEMORY_SCOPE_SYSTEM); }
__device__ __forceinline__ u32 lds_ld(const char LDS* p) { return __hip_atomic_load((const LDS u32*)p, __ATOMIC_RELAXED, __HIP_MEMORY_SCOPE_WORKGROUP); }
__device__ __forceinline__ void lds_st(char LDS* p, u32 v) { __hip_atomic_store((LDS u32*)p, v, __ATOMIC_RELAXED, __HIP_MEMORY_SCOPE_WORKGROUP); }

__device__ __forceinline__ bool give_up(char* myws, u32 call, u32 tmo, u64 t0) {
  bool bad = tmo == call || (__builtin_amdgcn_s_memrealtime() - t0) > DEADLINE;
  bad = __builtin_amdgcn_readfirstlane(bad);
  if (bad) st_agent32(myws + WS_TMO, call);
  return bad;
}
__device__ __forceinline__ float wave_max(float v) {
  int x = __float_as_int(v);
  x = max(x, __builtin_amdgcn_update_dpp(0, x, 0x111, 0xf, 0xf, false));
  x = max(x, __builtin_amdgcn_update_dpp(0, x, 0x112, 0xf, 0xf, false));
  x = max(x, __builtin_amdgcn_update_dpp(0, x, 0x114, 0xf, 0xf, false));
  x = max(x, __builtin_amdgcn_update_dpp(0, x, 0x118, 0xf, 0xf, false));
  int a = __builtin_amdgcn_readlane(x, 15), b = __builtin_amdgcn_readlane(x, 31), c = __builtin_amdgcn_readlane(x, 47), d = __builtin_amdgcn_readlane(x, 63);
  return __int_as_float(max(max(a, b), max(c, d)));
}
__device__ __forceinline__ u32 enc(u32 v) { return __builtin_elementwise_add_sat(v, 1u); }
__device__ __forceinline__ bool nz4(u32x4 v) { return v[0] != 0 && v[1] != 0 && v[2] != 0 && v[3] != 0; }
__device__ __forceinline__ bool has80(u32 v) { u32 t = v ^ 0x80808080u; return ((t - 0x01010101u) & ~t & 0x80808080u) != 0; }
__device__ __forceinline__ u32 q4(float a, float b, float c, float d, float inv) {
  int qa = (int)__builtin_amdgcn_fmed3f(__builtin_rintf(a * inv), -127.f, 127.f);
  int qb = (int)__builtin_amdgcn_fmed3f(__builtin_rintf(b * inv), -127.f, 127.f);
  int qc = (int)__builtin_amdgcn_fmed3f(__builtin_rintf(c * inv), -127.f, 127.f);
  int qd = (int)__builtin_amdgcn_fmed3f(__builtin_rintf(d * inv), -127.f, 127.f);
  return (qa & 0xff) | ((qb & 0xff) << 8) | ((qc & 0xff) << 16) | ((u32)qd << 24);
}
__device__ __forceinline__ u32 pack_bf16(float a, float b) {
  bf16x2 v = __builtin_convertvector(f32x2{a, b}, bf16x2);
  return __builtin_bit_cast(u32, v);
}
#define STAMP(i) do {} while (0)
constexpr int L_BASE = ((T - 4) % NS) * STAGE;
static_assert(((T - 3) % NS) == ((T - 4) % NS) + 1, "two adjacent dead slots");
constexpr int L_SCALE = L_BASE, L_STATE = L_BASE + 4  , L_TOKEN = L_BASE + 8, L_TILE = L_BASE + 1024, TROW = 528;
}

__global__ __launch_bounds__(512, 2) void fused_gemm_a2a_q8(Params P) {
#if __HIP_DEVICE_COMPILE__
  extern __shared__ __attribute__((aligned(16))) char smem_[];
  char LDS* const smem = (char LDS*)smem_;
  const int tid = threadIdx.x, lane = tid & 63;
  const int wid = __builtin_amdgcn_readfirstlane(tid >> 6);
  const int b = blockIdx.x, grp = b & 7, idx = b >> 3;
  const int kh = grp & 1, tn = (grp >> 1) * 8 + (idx & 7), tm = idx >> 3;
  const int partner = b ^ 1;
  const int n0 = tn * BN, m0 = tm * BM, k0 = kh * (K / KSPLIT);
  char* const myws = P.myws;
  STAMP(0);
  const int rrow = wid * 8 + (lane >> 3);
  const int in_off = (grp * MS + ((idx >> 2) & 3) * BM + (rrow >> 5) * 64 + (idx >> 4) * 32 + (rrow & 31)) * NSH + (idx & 3) * BN + (lane & 7) * 16;
  auto r_myout = __builtin_amdgcn_make_buffer_rsrc((void*)P.myout, 0, N * NSH, 0x00020000);
  const bool recv = grp != P.rank;
  u32 call;
  if (wid >= 4) {
    const int lw = wid - 4;
    u32 callv = ld_agent32(myws + WS_CALL);
    auto rw = __builtin_amdgcn_make_buffer_rsrc((void*)P.w, 0, K * N * 2, 0x00020000);
    auto rx = __builtin_amdgcn_make_buffer_rsrc((void*)P.x, 0, MS * K * 2, 0x00020000);
    int vw[4], vx[4];
#pragma unroll
    for (int j = 0; j < 4; ++j) {
      int k = 16 * lw + 4 * j + (lane >> 4);
      int F = (k & 3) | ((k >> 1) & 4);
      vw[j] = k * (N * 2) + (((lane & 15) ^ (F << 1)) * 16) + n0 * 2;
      int r = (lw * 4 + j) * 8 + (lane >> 3);
      vx[j] = (m0 + r) * (K * 2) + (((lane & 7) ^ ((r >> 1) & 7)) * 16);
    }
    const int rj = (tm + tn) & 3;
    auto issue = [&](int s, int slot_off) {
      const int ss = (s + ((s & 3) == rj ? 4 : 0) + grp * 4) & (T - 1);
      const int sw = (k0 + ss * BK) * (N * 2), sx = (k0 + ss * BK) * 2;
#pragma unroll
      for (int j = 0; j < 4; ++j) __builtin_amdgcn_raw_ptr_buffer_load_lds(rw, (LDS void*)(smem + slot_off + (lw * 4 + j) * 1024), 16, vw[j], sw, 0, 0);
#pragma unroll
      for (int j = 0; j < 4; ++j) __builtin_amdgcn_raw_ptr_buffer_load_lds(rx, (LDS void*)(smem + slot_off + WBYTES + (lw * 4 + j) * 1024), 16, vx[j], sx, 0, 0);
    };
#pragma unroll
    for (int s = 0; s < NS - 1; ++s) issue(s, s * STAGE);
    int slot = (NS - 1) * STAGE;
#pragma unroll 1
    for (int s = 0; s <= T - NS; ++s) {
      wait_vm<(NS - 2) * LPW>();
      bar();
      issue(s + NS - 1, slot);
      slot += STAGE; if (slot == NS * STAGE) slot = 0;
    }
    if (NS == 5) { wait_vm<3 * LPW>(); bar(); }
    wait_vm<2 * LPW>(); bar();
    wait_vm<LPW>(); bar();
    wait_vm<0>(); bar();
    call = __builtin_amdgcn_readfirstlane(callv) + 1;
    if (wid == 4 && lane == 0) { lds_st(smem + L_STATE, 0); lds_st(smem + L_TOKEN, 0); }
  } else {
    u32 callv = ld_agent32(myws + WS_CALL);
    if (recv) {
      __builtin_amdgcn_raw_buffer_store_b128(u32x4{0x80808080u, 0x80808080u, 0x80808080u, 0x80808080u}, r_myout, in_off, 0, AUX_SYS);
      __builtin_amdgcn_raw_buffer_store_b128(u32x4{0x80808080u, 0x80808080u, 0x80808080u, 0x80808080u}, r_myout, in_off, 64 * NSH, AUX_SYS);
    }
    f32x4 acc[4][4];
    const int wm = wid & 1, wn = wid >> 1;
    const int g = lane >> 4, q = (lane & 15) >> 2, p = lane & 3;
    const int F = q | ((g & 1) << 2);
    int aw[4], ax[2][2];
#pragma unroll
    for (int c = 0; c < 4; ++c) aw[c] = (8 * g + q) * 256 + ((((4 * wn + c) ^ F) << 1) | (p >> 1)) * 16 + 8 * (p & 1);
#pragma unroll
    for (int ks = 0; ks < 2; ++ks) {
      int r = wm * 64 + (lane & 15), a = WBYTES + r * 128 + (((4 * ks + g) ^ ((r >> 1) & 7)) * 16);
      ax[ks][0] = a + kh * 4096; ax[ks][1] = a + (1 - kh) * 4096;
    }
#pragma unroll
    for (int i = 0; i < 4; ++i)
#pragma unroll
      for (int j = 0; j < 4; ++j) acc[i][j] = f32x4{0, 0, 0, 0};
    s16x8 W0[4][2], X0[4][2], W1[4][2], X1[4][2];
    auto rd = [&](s16x8(&W)[4][2], s16x8(&X)[4][2], int so) {
#pragma unroll
      for (int ks = 0; ks < 2; ++ks) {
#pragma unroll
        for (int j = 0; j < 4; ++j) X[j][ks] = *(LDS s16x8*)(smem + so + ax[ks][j >> 1] + (j & 1) * 2048);
#pragma unroll
        for (int c = 0; c < 4; ++c) {
          s16x4 lo = __builtin_amdgcn_ds_read_tr16_b64_v4i16((LDS s16x4*)(smem + so + aw[c] + (32 * ks) * 256));
          s16x4 hi = __builtin_amdgcn_ds_read_tr16_b64_v4i16((LDS s16x4*)(smem + so + aw[c] + (32 * ks + 4) * 256));
          W[c][ks] = __builtin_shufflevector(lo, hi, 0, 1, 2, 3, 4, 5, 6, 7);
        }
      }
    };
    auto mm = [&](s16x8(&W)[4][2], s16x8(&X)[4][2]) {
#pragma unroll
      for (int ks = 0; ks < 2; ++ks)
#pragma unroll
        for (int c = 0; c < 4; ++c)
#pragma unroll
          for (int j = 0; j < 4; ++j)
            acc[c][j] = __builtin_amdgcn_mfma_f32_16x16x32_bf16(__builtin_bit_cast(bf16x8, W[c][ks]), __builtin_bit_cast(bf16x8, X[j][ks]), acc[c][j], 0, 0, 0);
    };
    auto sched = [&]() {
#pragma unroll
      for (int i = 0; i < 32; ++i) {
        __builtin_amdgcn_sched_group_barrier(0x8, 1, 0);
        if (i < 24) __builtin_amdgcn_sched_group_barrier(0x100, 1, 0);
      }
    };
    int slot = 0;
    bar();
    rd(W0, X0, slot); slot += STAGE;
    for (int s = 1; s < T - 1; s += 2) {
      bar_lgkm();
      rd(W1, X1, slot); mm(W0, X0); sched();
      slot += STAGE; if (slot == NS * STAGE) slot = 0;
      bar_lgkm();
      rd(W0, X0, slot); mm(W1, X1); sched();
      slot += STAGE; if (slot == NS * STAGE) slot = 0;
    }
    bar_lgkm();
    rd(W1, X1, slot); mm(W0, X0); sched();
#pragma unroll
    for (int jj = 2; jj >= 0; jj -= 2)
#pragma unroll
      for (int ks = 0; ks < 2; ++ks)
#pragma unroll
        for (int c = 0; c < 4; ++c)
#pragma unroll
          for (int j = jj; j < jj + 2; ++j)
            acc[c][j] = __builtin_amdgcn_mfma_f32_16x16x32_bf16(__builtin_bit_cast(bf16x8, W1[c][ks]), __builtin_bit_cast(bf16x8, X1[j][ks]), acc[c][j], 0, 0, 0);
    call = __builtin_amdgcn_readfirstlane(callv) + 1;
    STAMP(1);
    auto rs = __builtin_amdgcn_make_buffer_rsrc((void*)(myws + WS_SLAB), 0, NWG * 32768, 0x00020000);
    const int so_out = (partner * 4 + wid) * 8192, so_in = (b * 4 + wid) * 8192;
    const bool l2path = XMODE == 2;
    const u32 tag = call & 1;
#pragma unroll
    for (int c = 0; c < 4; ++c)
#pragma unroll
      for (int j = 0; j < 2; ++j) {
        u32x4 e = __builtin_bit_cast(u32x4, acc[c][2 + j]);
        e = (e & ~1u) | tag;
        if (l2path) __builtin_amdgcn_raw_buffer_store_b128(e, rs, lane * 16 + (c * 2 + j) * 1024, so_out, 0);
        else __builtin_amdgcn_raw_buffer_store_b128(e, rs, lane * 16 + (c * 2 + j) * 1024, so_out, AUX_SC1);
      }
    STAMP(2);
    u32x4 in[8];
    bool ok = true;
    {
      u64 t0 = __builtin_amdgcn_s_memrealtime();
      for (;;) {
        asm volatile("" ::: "memory");
#pragma unroll
        for (int i = 0; i < 8; ++i) in[i] = __builtin_amdgcn_raw_buffer_load_b128(rs, lane * 16 + i * 1024, so_in, AUX_SC1);
        u32 tm_ = ld_agent32(myws + WS_TMO);
        u32 an = ~0u, orr = 0;
#pragma unroll
        for (int i = 0; i < 8; ++i) { an &= (in[i][0] & in[i][1]) & (in[i][2] & in[i][3]); orr |= (in[i][0] | in[i][1]) | (in[i][2] | in[i][3]); }
        if (__all(((tag ? an : ~orr) & 1u) != 0)) break;
        if (give_up(myws, call, tm_, t0)) { ok = false; break; }
      }
    }
    STAMP(3);
    float am = 0.f;
#pragma unroll
    for (int c = 0; c < 4; ++c)
#pragma unroll
      for (int j = 0; j < 2; ++j) {
        acc[c][j] += __builtin_bit_cast(f32x4, in[c * 2 + j]);
#pragma unroll
        for (int r = 0; r < 4; ++r) am = fmaxf(am, fabsf(acc[c][j][r]));
      }
    am = wave_max(am);
    if (lane == 0 && ok) st_agent64(myws + WS_L1G + (size_t)((call & 1) * 8 + grp) * 1024 + (idx * 4 + wid) * 8, ((u64)call << 32) | __float_as_uint(am));
#pragma unroll
    for (int c = 0; c < 4; ++c)
#pragma unroll
      for (int j = 0; j < 2; ++j) {
        int row = wm * 32 + j * 16 + (lane & 15), col = wn * 64 + c * 16 + 4 * g;
        *(LDS f32x4*)(smem + L_TILE + row * TROW + col * 4) = acc[c][j];
      }
    STAMP(4);
  }
  bar_lgkm();
  const int par = call & 1;
  const int dest = tn >> 2;
  f32x4 yv[4];
#pragma unroll
  for (int i = 0; i < 4; ++i) yv[i] = *(LDS f32x4*)(smem + L_TILE + rrow * TROW + (lane & 7) * 64 + i * 16);
  auto r_stg_in = __builtin_amdgcn_make_buffer_rsrc((void*)(myws + WS_STG + par * STG_BYTES), 0, (int)STG_BYTES, 0x00020000);
  const int stg_in_off = ((grp * 32 + idx) * 8 + wid) * 2048 + lane * 16;
  const int stg_probe = ((grp * 32 + idx) * 8 + wid) * 2048 + 2044;
  bool sent = false, got = false;
  u32x4 e0, e1;
  u64 t0 = __builtin_amdgcn_s_memrealtime();
  const bool can_early = EARLY && dest != P.rank;
  auto send_early = [&]() {
    auto r_stg = __builtin_amdgcn_make_buffer_rsrc((void*)(P.ws[dest] + WS_STG + par * STG_BYTES), 0, (int)STG_BYTES, 0x00020000);
    const int so = ((P.rank * 32 + ((tn & 3) + 4 * tm + 16 * kh)) * 8 + wid) * 2048;
    u32x4 a = {enc(pack_bf16(yv[0][0], yv[0][1])), enc(pack_bf16(yv[0][2], yv[0][3])), enc(pack_bf16(yv[1][0], yv[1][1])), enc(pack_bf16(yv[1][2], yv[1][3]))};
    u32x4 c = {enc(pack_bf16(yv[2][0], yv[2][1])), enc(pack_bf16(yv[2][2], yv[2][3])), enc(pack_bf16(yv[3][0], yv[3][1])), enc(pack_bf16(yv[3][2], yv[3][3]))};
    __builtin_amdgcn_raw_buffer_store_b128(a, r_stg, lane * 16, so, AUX_SYS);
    __builtin_amdgcn_raw_buffer_store_b128(c, r_stg, lane * 16 + 1024, so, AUX_SYS);
    sent = true;
    STAMP(9);
  };
  if (wid == 4) {
    const char* gsrc = myws + WS_GG + (size_t)((par * REP + (idx & (REP - 1))) * 64 + lane) * 8;
    u64 v; bool ok = true;
    for (;;) {
      v = ld_sys64(gsrc);
      u32 tm_ = ld_agent32(myws + WS_TMO);
      if (__all((u32)(v >> 32) == call)) { ok = tm_ != call; break; }
      if (give_up(myws, call, tm_, t0)) { ok = false; break; }
    }
    float gm = wave_max(__uint_as_float((u32)v));
    if (lane == 0) {
      lds_st(smem + L_SCALE, __float_as_uint(gm > 0.f ? 127.0f / gm : 0.f));
      __builtin_amdgcn_s_waitcnt(0xC07F);
      lds_st(smem + L_STATE, ok ? 1u : 2u);
      if (b == 0) st_agent32(myws + WS_CALL, call);
    }
    STAMP(6);
  } else {
    if (wid == 5 && idx == 0) {
      const char* l1 = myws + WS_L1G + (size_t)(par * 8 + grp) * 1024 + lane * 16;
      char* dst = P.ws[lane & 7] + WS_GG + (size_t)((par * REP + (lane >> 3)) * 64 + P.rank * 8 + grp) * 8;
      auto rl = __builtin_amdgcn_make_buffer_rsrc((void*)myws, 0, 1 << 20, 0x00020000);
      for (;;) {
        asm volatile("" ::: "memory");
        u32x4 v = __builtin_amdgcn_raw_buffer_load_b128(rl, (int)(l1 - myws), 0, AUX_SC1);
        u32 tm_ = ld_agent32(myws + WS_TMO);
        if (__all(v[1] == call && v[3] == call)) {
          float gm = wave_max(fmaxf(__uint_as_float(v[0]), __uint_as_float(v[2])));
          st_sys64(dst, ((u64)call << 32) | __float_as_uint(gm));
          break;
        }
        if (give_up(myws, call, tm_, t0)) break;
      }
      STAMP(5);
    }
    const u64 due = t0 + (u64)(wid - (wid > 4)) * PACE;
    while (__builtin_amdgcn_readfirstlane(lds_ld(smem + L_STATE)) == 0) {
      if (can_early && !sent && __builtin_amdgcn_s_memrealtime() >= due) {
        send_early();
      } else if (EARLY && recv && !got && (sent || !can_early)) {
        asm volatile("" ::: "memory");
        if (__builtin_amdgcn_readfirstlane(__builtin_amdgcn_raw_buffer_load_b32(r_stg_in, stg_probe, 0, AUX_SYS)) != 0) {
          e0 = __builtin_amdgcn_raw_buffer_load_b128(r_stg_in, stg_in_off, 0, AUX_SYS);
          e1 = __builtin_amdgcn_raw_buffer_load_b128(r_stg_in, stg_in_off + 1024, 0, AUX_SYS);
          got = __all(nz4(e0) && nz4(e1));
        } else __builtin_amdgcn_s_sleep(PRE_SLEEP);
      } else {
        __builtin_amdgcn_s_sleep(1);
      }
    }
  }
  __builtin_amdgcn_s_waitcnt(0xC07F);
  const u32 state = __builtin_amdgcn_readfirstlane(lds_ld(smem + L_STATE));
  const float inv = __uint_as_float(lds_ld(smem + L_SCALE));
  if (state == 1) {
    if (!sent) {
      u32x4 o = {q4(yv[0][0], yv[0][1], yv[0][2], yv[0][3], inv), q4(yv[1][0], yv[1][1], yv[1][2], yv[1][3], inv),
                 q4(yv[2][0], yv[2][1], yv[2][2], yv[2][3], inv), q4(yv[3][0], yv[3][1], yv[3][2], yv[3][3], inv)};
      const int grow = P.rank * MS + m0 + (rrow >> 5) * 64 + kh * 32 + (rrow & 31);
      auto ro = __builtin_amdgcn_make_buffer_rsrc((void*)P.out[dest], 0, N * NSH, 0x00020000);
      __builtin_amdgcn_raw_buffer_store_b128(o, ro, grow * NSH + (tn & 3) * BN + (lane & 7) * 16, 0, AUX_SYS);
    }
    STAMP(7);
    if (recv) {
      while (!got) {
        asm volatile("" ::: "memory");
        u32x4 o = __builtin_amdgcn_raw_buffer_load_b128(r_myout, in_off, 0, AUX_SYS);
        if (EARLY) {
          e0 = __builtin_amdgcn_raw_buffer_load_b128(r_stg_in, stg_in_off, 0, AUX_SYS);
          e1 = __builtin_amdgcn_raw_buffer_load_b128(r_stg_in, stg_in_off + 1024, 0, AUX_SYS);
        }
        u32 tm_ = ld_agent32(myws + WS_TMO);
        if (__all(!(has80(o[0]) || has80(o[1]) || has80(o[2]) || has80(o[3])))) break;
        if (EARLY) got = __all(nz4(e0) && nz4(e1));
        if (!got && give_up(myws, call, tm_, t0)) break;
        if (!got) __builtin_amdgcn_s_sleep(FIN_SLEEP);
      }
      if (EARLY && got) {
        e0 -= 1u; e1 -= 1u;
        auto lo = [](u32 v) { return __uint_as_float(v << 16); };
        auto hi = [](u32 v) { return __uint_as_float(v & 0xffff0000u); };
        u32x4 o = {q4(lo(e0[0]), hi(e0[0]), lo(e0[1]), hi(e0[1]), inv), q4(lo(e0[2]), hi(e0[2]), lo(e0[3]), hi(e0[3]), inv),
                   q4(lo(e1[0]), hi(e1[0]), lo(e1[1]), hi(e1[1]), inv), q4(lo(e1[2]), hi(e1[2]), lo(e1[3]), hi(e1[3]), inv)};
        __builtin_amdgcn_raw_buffer_store_b128(o, r_myout, in_off, 0, AUX_SYS);
        __builtin_amdgcn_raw_buffer_store_b128(u32x4{0, 0, 0, 0}, r_stg_in, stg_in_off, 0, AUX_SYS);
        __builtin_amdgcn_raw_buffer_store_b128(u32x4{0, 0, 0, 0}, r_stg_in, stg_in_off + 1024, 0, AUX_SYS);
      }
      STAMP(8);
    }
  } else if (!recv) {
    const int grow = P.rank * MS + m0 + (rrow >> 5) * 64 + kh * 32 + (rrow & 31);
    if (dest == P.rank) __builtin_amdgcn_raw_buffer_store_b128(u32x4{0x80808080u, 0x80808080u, 0x80808080u, 0x80808080u}, r_myout, grow * NSH + (tn & 3) * BN + (lane & 7) * 16, 0, AUX_SYS);
  }
#endif
}

extern "C" void kernel_launch(int rank, int nranks, ncclComm_t comm, hipStream_t stream, void* const* d_in, const int* in_sizes, int n_in,
                              void* d_out, int out_size, const int* in_dtypes, int out_dtype, void* const* d_out_all,
                              void* const* d_ws_all, size_t ws_bytes) {
  if (nranks != NR || (unsigned)rank >= (unsigned)NR || ws_bytes < WS_NEED || n_in < 2 || in_sizes[0] != MS * K || in_sizes[1] != K * N || out_size != N * NSH) {
    fprintf(stderr, "unexpected problem shape\n");
    return;
  }
  Params P;
  P.x = (const u16*)d_in[0];
  P.w = (const u16*)d_in[1];
  for (int p = 0; p < NR; ++p) { P.out[p] = (char*)d_out_all[p]; P.ws[p] = (char*)d_ws_all[p]; }
  P.myws = P.ws[rank]; P.myout = (char*)d_out;
  P.rank = rank;
  hipError_t e = hipFuncSetAttribute((const void*)fused_gemm_a2a_q8, hipFuncAttributeMaxDynamicSharedMemorySize, LDS_BYTES);
  if (e != hipSuccess) { fprintf(stderr, "hipFuncSetAttribute: %s\n", hipGetErrorString(e)); return; }
  hipLaunchKernelGGL(fused_gemm_a2a_q8, dim3(NWG), dim3(512), LDS_BYTES, stream, P);
}
```

```cpp
#include <hip/hip_runtime.h>
#include <rccl/rccl.h>
#include <cstdio>

typedef unsigned short u16;
typedef unsigned int u32;
typedef unsigned long long u64;
typedef __attribute__((ext_vector_type(8))) short s16x8;
typedef __attribute__((ext_vector_type(4))) short s16x4;
typedef __attribute__((ext_vector_type(8))) __bf16 bf16x8;
typedef __attribute__((ext_vector_type(2))) __bf16 bf16x2;
typedef __attribute__((ext_vector_type(4))) float f32x4;
typedef __attribute__((ext_vector_type(2))) float f32x2;
typedef __attribute__((ext_vector_type(2))) _Float16 f16x2;
typedef __attribute__((ext_vector_type(4))) unsigned int u32x4;
#define LDS __attribute__((address_space(3)))
#define GAS __attribute__((address_space(1)))

namespace {
constexpr int EARLY = 1;
constexpr int FIN_SLEEP = 2;
constexpr int PACE = 160;
constexpr int PRE_SLEEP = 4;
constexpr int NR = 8;
constexpr int MS = 512, N = 4096, K = 4096;
constexpr int NSH = N / NR;
constexpr int BM = 128, BN = 128, BK = 64, KSPLIT = 2, NS = 4;
constexpr int WBYTES = BK * BN * 2, XBYTES = BM * BK * 2, STAGE = WBYTES + XBYTES;
constexpr int T = K / KSPLIT / BK;
constexpr int NWG = 256;
constexpr int LDS_BYTES = NS * STAGE;
constexpr int LPW = 8;
constexpr size_t WS_CALL = 0;
constexpr size_t WS_TMO = 128;
constexpr size_t WS_L1G = 4096;
constexpr int REP = 8;
constexpr size_t WS_GG = WS_L1G + 2 * 8 * 128 * 8;
constexpr size_t WS_DBG = 512 * 1024;
constexpr size_t WS_SLAB = 1 << 20;
constexpr size_t WS_STG = WS_SLAB + (size_t)NWG * 32768;
constexpr size_t STG_BYTES = (size_t)N * NSH * 2;
constexpr size_t WS_NEED = WS_STG + 2 * STG_BYTES;
constexpr u64 DEADLINE = 100000000ull;
constexpr int AUX_SC1 = 16, AUX_SYS = 17;

struct Params {
  const u16* x; const u16* w;
  char* myws; char* myout;
  char* out[NR]; char* ws[NR];
  int rank;
};

template <int N_> __device__ __forceinline__ void wait_vm() { asm volatile("s_waitcnt vmcnt(%0)" ::"i"(N_) : "memory"); }
__device__ __forceinline__ void bar() { asm volatile("s_barrier" ::: "memory"); }
__device__ __forceinline__ void bar_lgkm() {
  __builtin_amdgcn_sched_barrier(0);
  __builtin_amdgcn_s_waitcnt(0xC07F);
  asm volatile("s_barrier" ::: "memory");
  __builtin_amdgcn_sched_barrier(0);
}
__device__ __forceinline__ u32 ld_agent32(const char* p) { return __hip_atomic_load((const GAS u32*)p, __ATOMIC_RELAXED, __HIP_MEMORY_SCOPE_AGENT); }
__device__ __forceinline__ u64 ld_sys64(const char* p) { return __hip_atomic_load((const GAS u64*)p, __ATOMIC_RELAXED, __HIP_MEMORY_SCOPE_SYSTEM); }
__device__ __forceinline__ void st_agent32(char* p, u32 v) { __hip_atomic_store((GAS u32*)p, v, __ATOMIC_RELAXED, __HIP_MEMORY_SCOPE_AGENT); }
__device__ __forceinline__ void st_agent64(char* p, u64 v) { __hip_atomic_store((GAS u64*)p, v, __ATOMIC_RELAXED, __HIP_MEMORY_SCOPE_AGENT); }
__device__ __forceinline__ void st_sys64(char* p, u64 v) { __hip_atomic_store((GAS u64*)p, v, __ATOMIC_RELAXED, __HIP_MEMORY_SCOPE_SYSTEM); }
__device__ __forceinline__ u32 lds_ld(const char LDS* p) { return __hip_atomic_load((const LDS u32*)p, __ATOMIC_RELAXED, __HIP_MEMORY_SCOPE_WORKGROUP); }
__device__ __forceinline__ void lds_st(char LDS* p, u32 v) { __hip_atomic_store((LDS u32*)p, v, __ATOMIC_RELAXED, __HIP_MEMORY_SCOPE_WORKGROUP); }

__device__ __forceinline__ bool give_up(char* myws, u32 call, u32 tmo, u64 t0) {
  bool bad = tmo == call || (__builtin_amdgcn_s_memrealtime() - t0) > DEADLINE;
  bad = __builtin_amdgcn_readfirstlane(bad);
  if (bad) st_agent32(myws + WS_TMO, call);
  return bad;
}
__device__ __forceinline__ float wave_max(float v) {
  int x = __float_as_int(v);
  x = max(x, __builtin_amdgcn_update_dpp(0, x, 0x111, 0xf, 0xf, false));
  x = max(x, __builtin_amdgcn_update_dpp(0, x, 0x112, 0xf, 0xf, false));
  x = max(x, __builtin_amdgcn_update_dpp(0, x, 0x114, 0xf, 0xf, false));
  x = max(x, __builtin_amdgcn_update_dpp(0, x, 0x118, 0xf, 0xf, false));
  int a = __builtin_amdgcn_readlane(x, 15), b = __builtin_amdgcn_readlane(x, 31), c = __builtin_amdgcn_readlane(x, 47), d = __builtin_amdgcn_readlane(x, 63);
  return __int_as_float(max(max(a, b), max(c, d)));
}
template <int AUX, typename R> __device__ __forceinline__ void st128(u32x4 v, R rsrc, int voff) { __builtin_amdgcn_raw_buffer_store_b128(v, rsrc, voff, 0, AUX); }
__device__ __forceinline__ u32 enc(u32 v) { return __builtin_elementwise_add_sat(v, 1u); }
__device__ __forceinline__ bool nz4(u32x4 v) { return v[0] != 0 && v[1] != 0 && v[2] != 0 && v[3] != 0; }
__device__ __forceinline__ bool has80(u32 v) { u32 t = v ^ 0x80808080u; return ((t - 0x01010101u) & ~t & 0x80808080u) != 0; }
__device__ __forceinline__ u32 q4(float a, float b, float c, float d, float inv) {
  const float M = 12582912.0f;
  u32 qa = __float_as_uint(__builtin_fmaf(a, inv, M)), qb = __float_as_uint(__builtin_fmaf(b, inv, M));
  u32 qc = __float_as_uint(__builtin_fmaf(c, inv, M)), qd = __float_as_uint(__builtin_fmaf(d, inv, M));
  u32 lo = __builtin_amdgcn_perm(qb, qa, 0x0c0c0400u), hi = __builtin_amdgcn_perm(qd, qc, 0x04000c0cu);
  return lo | hi;
}
__device__ __forceinline__ u32 pack_bf16(float a, float b) {
  bf16x2 v = __builtin_convertvector(f32x2{a, b}, bf16x2);
  return __builtin_bit_cast(u32, v);
}
#define STAMP(i) do {} while (0)
constexpr int L_BASE = ((T - 4) % NS) * STAGE;
static_assert(((T - 3) % NS) == ((T - 4) % NS) + 1, "two adjacent dead slots");
static_assert(NS == 4 && LPW == 8, "counted waits of the loader loop");
constexpr int L_SCALE = L_BASE, L_STATE = L_BASE + 4  , L_TILE = L_BASE + 1024, TROW = 528;
}

__global__ __launch_bounds__(512, 2) void fused_gemm_a2a_q8(Params P) {
#if __HIP_DEVICE_COMPILE__
  extern __shared__ __attribute__((aligned(16))) char smem_[];
  char LDS* const smem = (char LDS*)smem_;
  const int tid = threadIdx.x, lane = tid & 63;
  const int wid = __builtin_amdgcn_readfirstlane(tid >> 6);
  const int b = blockIdx.x, grp = b & 7, idx = b >> 3;
  const int kh = grp & 1, tn = (grp >> 1) * 8 + (idx & 7), tm = idx >> 3;
  const int partner = b ^ 1;
  const int n0 = tn * BN, m0 = tm * BM, k0 = kh * (K / KSPLIT);
  const int dest = tn >> 2;
  const u16* xp = P.x; const u16* wp = P.w;
  char* myws = P.myws; char* myout = P.myout; int rank = P.rank;
  char* out_dest = P.out[dest]; char* ws_dest = P.ws[dest];
  asm volatile("" : "+s"(xp), "+s"(wp), "+s"(myws), "+s"(myout), "+s"(rank), "+s"(out_dest), "+s"(ws_dest));
  STAMP(0);
  const int rrow = wid * 8 + (lane >> 3);
  const int in_off = (grp * MS + ((idx >> 2) & 3) * BM + (rrow >> 5) * 64 + (idx >> 4) * 32 + (rrow & 31)) * NSH + (idx & 3) * BN + (lane & 7) * 16;
  auto r_myout = __builtin_amdgcn_make_buffer_rsrc((void*)myout, 0, N * NSH, 0x00020000);
  const bool recv = grp != rank;
  u32 call;
  char* ws_lane_all = nullptr;
  if (wid >= 4) {
    const int lw = wid - 4;
    u32 callv = ld_agent32(myws + WS_CALL);
    char* ws_lane = P.ws[lane & 7];
    auto rw = __builtin_amdgcn_make_buffer_rsrc((void*)wp, 0, K * N * 2, 0x00020000);
    auto rx = __builtin_amdgcn_make_buffer_rsrc((void*)xp, 0, MS * K * 2, 0x00020000);
    int vw[4], vx[4];
#pragma unroll
    for (int j = 0; j < 4; ++j) {
      int k = 16 * lw + 4 * j + (lane >> 4);
      int F = (k & 3) | ((k >> 1) & 4);
      vw[j] = k * (N * 2) + (((lane & 15) ^ (F << 1)) * 16) + n0 * 2;
      int r = (lw * 4 + j) * 8 + (lane >> 3);
      vx[j] = (m0 + r) * (K * 2) + (((lane & 7) ^ ((r >> 1) & 7)) * 16);
    }
    const int rj = (tm + tn) & 3;
    auto issue = [&](int s, int slot_off) {
      const int ss = (s + ((s & 3) == rj ? 4 : 0) + grp * 4) & (T - 1);
      const int sw = (k0 + ss * BK) * (N * 2), sx = (k0 + ss * BK) * 2;
#pragma unroll
      for (int j = 0; j < 4; ++j) __builtin_amdgcn_raw_ptr_buffer_load_lds(rw, (LDS void*)(smem + slot_off + (lw * 4 + j) * 1024), 16, vw[j], sw, 0, 0);
#pragma unroll
      for (int j = 0; j < 4; ++j) __builtin_amdgcn_raw_ptr_buffer_load_lds(rx, (LDS void*)(smem + slot_off + WBYTES + (lw * 4 + j) * 1024), 16, vx[j], sx, 0, 0);
    };
#pragma unroll
    for (int s = 0; s < NS - 1; ++s) issue(s, s * STAGE);
    int slot = (NS - 1) * STAGE;
    auto next = [&]() { slot += STAGE; if (slot == NS * STAGE) slot = 0; };
#pragma unroll 1
    for (int s = 0; s + 3 < T; ++s) { wait_vm<16>(); bar(); issue(s + 3, slot); next(); }
    wait_vm<16>(); bar(); wait_vm<8>(); bar(); wait_vm<0>(); bar();
    call = __builtin_amdgcn_readfirstlane(callv) + 1;
    ws_lane_all = ws_lane;
    if (wid == 4 && lane == 0) lds_st(smem + L_STATE, 0);
  } else {
    u32 callv = ld_agent32(myws + WS_CALL);
    if (recv) {
      st128<AUX_SYS>(u32x4{0x80808080u, 0x80808080u, 0x80808080u, 0x80808080u}, r_myout, in_off);
      st128<AUX_SYS>(u32x4{0x80808080u, 0x80808080u, 0x80808080u, 0x80808080u}, r_myout, in_off + 64 * NSH);
    }
    f32x4 acc[4][4];
    const int wm = wid & 1, wn = wid >> 1;
    const int g = lane >> 4, q = (lane & 15) >> 2, p = lane & 3;
    const int F = q | ((g & 1) << 2);
    int aw[4], ax[2][2];
#pragma unroll
    for (int c = 0; c < 4; ++c) aw[c] = (8 * g + q) * 256 + ((((4 * wn + c) ^ F) << 1) | (p >> 1)) * 16 + 8 * (p & 1);
#pragma unroll
    for (int ks = 0; ks < 2; ++ks) {
      int r = wm * 64 + (lane & 15), a = WBYTES + r * 128 + (((4 * ks + g) ^ ((r >> 1) & 7)) * 16);
      ax[ks][0] = a + kh * 4096; ax[ks][1] = a + (1 - kh) * 4096;
    }
#pragma unroll
    for (int i = 0; i < 4; ++i)
#pragma unroll
      for (int j = 0; j < 4; ++j) acc[i][j] = f32x4{0, 0, 0, 0};
    s16x8 W0[4][2], X0[4][2], W1[4][2], X1[4][2];
    auto rd = [&](s16x8(&W)[4][2], s16x8(&X)[4][2], int so) {
#pragma unroll
      for (int ks = 0; ks < 2; ++ks) {
#pragma unroll
        for (int j = 0; j < 4; ++j) X[j][ks] = *(LDS s16x8*)(smem + so + ax[ks][j >> 1] + (j & 1) * 2048);
#pragma unroll
        for (int c = 0; c < 4; ++c) {
          s16x4 lo = __builtin_amdgcn_ds_read_tr16_b64_v4i16((LDS s16x4*)(smem + so + aw[c] + (32 * ks) * 256));
          s16x4 hi = __builtin_amdgcn_ds_read_tr16_b64_v4i16((LDS s16x4*)(smem + so + aw[c] + (32 * ks + 4) * 256));
          W[c][ks] = __builtin_shufflevector(lo, hi, 0, 1, 2, 3, 4, 5, 6, 7);
        }
      }
    };
    auto mm = [&](s16x8(&W)[4][2], s16x8(&X)[4][2]) {
#pragma unroll
      for (int ks = 0; ks < 2; ++ks)
#pragma unroll
        for (int c = 0; c < 4; ++c)
#pragma unroll
          for (int j = 0; j < 4; ++j)
            acc[c][j] = __builtin_amdgcn_mfma_f32_16x16x32_bf16(__builtin_bit_cast(bf16x8, W[c][ks]), __builtin_bit_cast(bf16x8, X[j][ks]), acc[c][j], 0, 0, 0);
    };
    auto sched = [&]() {
#pragma unroll
      for (int i = 0; i < 32; ++i) {
        __builtin_amdgcn_sched_group_barrier(0x8, 1, 0);
        if (i < 24) __builtin_amdgcn_sched_group_barrier(0x100, 1, 0);
      }
    };
    auto mmj = [&](s16x8(&W)[4][2], s16x8(&X)[4][2], int j0) {
#pragma unroll
      for (int ks = 0; ks < 2; ++ks)
#pragma unroll
        for (int c = 0; c < 4; ++c)
#pragma unroll
          for (int j = j0; j < j0 + 2; ++j)
            acc[c][j] = __builtin_amdgcn_mfma_f32_16x16x32_bf16(__builtin_bit_cast(bf16x8, W[c][ks]), __builtin_bit_cast(bf16x8, X[j][ks]), acc[c][j], 0, 0, 0);
    };
    auto rs = __builtin_amdgcn_make_buffer_rsrc((void*)(myws + WS_SLAB), 0, NWG * 32768, 0x00020000);
    const int so_out = (partner * 4 + wid) * 8192, so_in = (b * 4 + wid) * 8192;
    u32 tag;
    constexpr int NX = 4;
    u32 in_sc;
    auto send_partial = [&]() {
      call = __builtin_amdgcn_readfirstlane(callv) + 1;
      tag = call & 1;
      float pm = 0.f;
#pragma unroll
      for (int c = 0; c < 4; ++c)
#pragma unroll
        for (int j = 2; j < 4; ++j)
#pragma unroll
          for (int r = 0; r < 4; ++r) pm = fmaxf(pm, fabsf(acc[c][j][r]));
      u32 eb = __float_as_uint(pm) & 0x7f800000u;
      eb = min(max(eb, 15u << 23), 254u << 23);
      const float up = __uint_as_float((268u << 23) - eb);
      const u32 dn = eb - (14u << 23);
#pragma unroll
      for (int c = 0; c < 4; ++c) {
        u32x4 e;
#pragma unroll
        for (int j = 0; j < 2; ++j)
#pragma unroll
          for (int r = 0; r < 2; ++r) {
            f16x2 h = __builtin_convertvector(f32x2{acc[c][2 + j][2 * r] * up, acc[c][2 + j][2 * r + 1] * up}, f16x2);
            e[j * 2 + r] = (__builtin_bit_cast(u32, h) & ~1u) | tag;
          }
        st128<AUX_SC1>(e, rs, lane * 16 + c * 1024 + so_out);
      }
      __builtin_amdgcn_raw_buffer_store_b32(dn | tag, rs, lane * 4 + 4096, so_out, AUX_SC1);
      STAMP(2);
    };
    u32x4 in[NX];
    auto load_partial = [&]() {
#pragma unroll
      for (int i = 0; i < NX; ++i) in[i] = __builtin_amdgcn_raw_buffer_load_b128(rs, lane * 16 + i * 1024, so_in, AUX_SC1);
      in_sc = __builtin_amdgcn_raw_buffer_load_b32(rs, lane * 4 + 4096, so_in, AUX_SC1);
    };
    int slot = 0;
    auto next = [&]() { slot += STAGE; if (slot == NS * STAGE) slot = 0; };
    bar();
    rd(W0, X0, slot); next();
    for (int s = 1; s < T - 1; s += 2) {
      bar_lgkm();
      rd(W1, X1, slot); mm(W0, X0); sched(); next();
      bar_lgkm();
      rd(W0, X0, slot); mm(W1, X1); sched(); next();
    }
    bar_lgkm();
    rd(W1, X1, slot); mm(W0, X0); sched(); next();
    mmj(W1, X1, 2); send_partial(); mmj(W1, X1, 0);
    load_partial();
    STAMP(1);
    bool ok = true;
    {
      u64 t0 = __builtin_amdgcn_s_memrealtime();
      u32 tm_ = 0;
      for (;;) {
        u32 an = in_sc, orr = in_sc;
#pragma unroll
        for (int i = 0; i < NX; ++i) { an &= (in[i][0] & in[i][1]) & (in[i][2] & in[i][3]); orr |= (in[i][0] | in[i][1]) | (in[i][2] | in[i][3]); }
        if (__all(((tag ? an : ~orr) & 1u) != 0)) break;
        if (give_up(myws, call, tm_, t0)) { ok = false; break; }
        asm volatile("" ::: "memory");
        load_partial();
        tm_ = ld_agent32(myws + WS_TMO);
      }
    }
    STAMP(3);
    float am = 0.f;
#pragma unroll
    for (int c = 0; c < 4; ++c)
#pragma unroll
      for (int j = 0; j < 2; ++j) {
        const float dn = __uint_as_float(in_sc & ~1u);
#pragma unroll
        for (int r = 0; r < 2; ++r) {
          float a0 = acc[c][j][2 * r], a1 = acc[c][j][2 * r + 1];
          asm("v_fma_mix_f32 %0, %1, %2, %0 op_sel_hi:[1,0,0]" : "+v"(a0) : "v"(in[c][j * 2 + r]), "v"(dn));
          asm("v_fma_mix_f32 %0, %1, %2, %0 op_sel:[1,0,0] op_sel_hi:[1,0,0]" : "+v"(a1) : "v"(in[c][j * 2 + r]), "v"(dn));
          acc[c][j][2 * r] = a0; acc[c][j][2 * r + 1] = a1;
        }
#pragma unroll
        for (int r = 0; r < 4; ++r) am = fmaxf(am, fabsf(acc[c][j][r]));
      }
    am = wave_max(am);
    if (lane == 0 && ok) st_agent64(myws + WS_L1G + (size_t)((call & 1) * 8 + grp) * 1024 + (idx * 4 + wid) * 8, ((u64)call << 32) | __float_as_uint(am));
#pragma unroll
    for (int c = 0; c < 4; ++c)
#pragma unroll
      for (int j = 0; j < 2; ++j) {
        int row = wm * 32 + j * 16 + (lane & 15), col = wn * 64 + c * 16 + 4 * g;
        *(LDS f32x4*)(smem + L_TILE + row * TROW + col * 4) = acc[c][j];
      }
    STAMP(4);
  }
  bar_lgkm();
  const int par = call & 1;
  f32x4 yv[4];
#pragma unroll
  for (int i = 0; i < 4; ++i) yv[i] = *(LDS f32x4*)(smem + L_TILE + rrow * TROW + (lane & 7) * 64 + i * 16);
  auto r_stg_in = __builtin_amdgcn_make_buffer_rsrc((void*)(myws + WS_STG + par * STG_BYTES), 0, (int)STG_BYTES, 0x00020000);
  const int stg_in_off = ((grp * 32 + idx) * 8 + wid) * 2048 + lane * 16;
  const int stg_probe = ((grp * 32 + idx) * 8 + wid) * 2048 + 2044;
  bool sent = false, got = false;
  u32x4 e0, e1;
  u64 t0 = __builtin_amdgcn_s_memrealtime();
  const bool can_early = EARLY && dest != rank;
  auto send_early = [&]() {
    auto r_stg = __builtin_amdgcn_make_buffer_rsrc((void*)(ws_dest + WS_STG + par * STG_BYTES), 0, (int)STG_BYTES, 0x00020000);
    const int so = ((rank * 32 + ((tn & 3) + 4 * tm + 16 * kh)) * 8 + wid) * 2048;
    u32x4 a = {enc(pack_bf16(yv[0][0], yv[0][1])), enc(pack_bf16(yv[0][2], yv[0][3])), enc(pack_bf16(yv[1][0], yv[1][1])), enc(pack_bf16(yv[1][2], yv[1][3]))};
    u32x4 c = {enc(pack_bf16(yv[2][0], yv[2][1])), enc(pack_bf16(yv[2][2], yv[2][3])), enc(pack_bf16(yv[3][0], yv[3][1])), enc(pack_bf16(yv[3][2], yv[3][3]))};
    st128<AUX_SYS>(a, r_stg, lane * 16 + so);
    st128<AUX_SYS>(c, r_stg, lane * 16 + 1024 + so);
    sent = true;
    STAMP(9);
  };
  if (wid == 4) {
    const char* gsrc = myws + WS_GG + (size_t)((par * REP + (idx & (REP - 1))) * 64 + lane) * 8;
    u64 v; bool ok = true;
    for (;;) {
      v = ld_sys64(gsrc);
      u32 tm_ = ld_agent32(myws + WS_TMO);
      if (__all((u32)(v >> 32) == call)) { ok = tm_ != call; break; }
      if (give_up(myws, call, tm_, t0)) { ok = false; break; }
    }
    float gm = wave_max(__uint_as_float((u32)v));
    if (lane == 0) {
      lds_st(smem + L_SCALE, __float_as_uint(gm > 0.f ? 127.0f / gm : 0.f));
      __builtin_amdgcn_s_waitcnt(0xC07F);
      lds_st(smem + L_STATE, ok ? 1u : 2u);
      if (b == 0) st_agent32(myws + WS_CALL, call);
    }
    STAMP(6);
  } else {
    if (wid == 5 && idx == 0) {
      const char* l1 = myws + WS_L1G + (size_t)(par * 8 + grp) * 1024 + lane * 16;
      char* dst = ws_lane_all + WS_GG + (size_t)((par * REP + (lane >> 3)) * 64 + rank * 8 + grp) * 8;
      auto rl = __builtin_amdgcn_make_buffer_rsrc((void*)myws, 0, 1 << 20, 0x00020000);
      for (;;) {
        asm volatile("" ::: "memory");
        u32x4 v = __builtin_amdgcn_raw_buffer_load_b128(rl, (int)(l1 - myws), 0, AUX_SC1);
        u32 tm_ = ld_agent32(myws + WS_TMO);
        if (__all(v[1] == call && v[3] == call)) {
          float gm = wave_max(fmaxf(__uint_as_float(v[0]), __uint_as_float(v[2])));
          st_sys64(dst, ((u64)call << 32) | __float_as_uint(gm));
          break;
        }
        if (give_up(myws, call, tm_, t0)) break;
      }
      STAMP(5);
    }
    const u64 due = t0 + (u64)(wid - (wid > 4)) * PACE;
    while (__builtin_amdgcn_readfirstlane(lds_ld(smem + L_STATE)) == 0) {
      if (can_early && !sent && __builtin_amdgcn_s_memrealtime() >= due) {
        send_early();
      } else if (EARLY && recv && !got && (sent || !can_early)) {
        asm volatile("" ::: "memory");
        if (__builtin_amdgcn_readfirstlane(__builtin_amdgcn_raw_buffer_load_b32(r_stg_in, stg_probe, 0, AUX_SYS)) != 0) {
          e0 = __builtin_amdgcn_raw_buffer_load_b128(r_stg_in, stg_in_off, 0, AUX_SYS);
          e1 = __builtin_amdgcn_raw_buffer_load_b128(r_stg_in, stg_in_off + 1024, 0, AUX_SYS);
          got = __all(nz4(e0) && nz4(e1));
        } else __builtin_amdgcn_s_sleep(PRE_SLEEP);
      } else {
        __builtin_amdgcn_s_sleep(1);
      }
    }
  }
  __builtin_amdgcn_s_waitcnt(0xC07F);
  const u32 state = __builtin_amdgcn_readfirstlane(lds_ld(smem + L_STATE));
  const float inv = __uint_as_float(lds_ld(smem + L_SCALE));
  if (state == 1) {
    if (!sent) {
      u32x4 o = {q4(yv[0][0], yv[0][1], yv[0][2], yv[0][3], inv), q4(yv[1][0], yv[1][1], yv[1][2], yv[1][3], inv),
                 q4(yv[2][0], yv[2][1], yv[2][2], yv[2][3], inv), q4(yv[3][0], yv[3][1], yv[3][2], yv[3][3], inv)};
      const int grow = rank * MS + m0 + (rrow >> 5) * 64 + kh * 32 + (rrow & 31);
      auto ro = __builtin_amdgcn_make_buffer_rsrc((void*)out_dest, 0, N * NSH, 0x00020000);
      st128<AUX_SYS>(o, ro, grow * NSH + (tn & 3) * BN + (lane & 7) * 16);
    }
    STAMP(7);
    if (recv) {
      while (!got) {
        asm volatile("" ::: "memory");
        u32x4 o = __builtin_amdgcn_raw_buffer_load_b128(r_myout, in_off, 0, AUX_SYS);
        if (EARLY) {
          e0 = __builtin_amdgcn_raw_buffer_load_b128(r_stg_in, stg_in_off, 0, AUX_SYS);
          e1 = __builtin_amdgcn_raw_buffer_load_b128(r_stg_in, stg_in_off + 1024, 0, AUX_SYS);
        }
        u32 tm_ = ld_agent32(myws + WS_TMO);
        if (__all(!(has80(o[0]) || has80(o[1]) || has80(o[2]) || has80(o[3])))) break;
        if (EARLY) got = __all(nz4(e0) && nz4(e1));
        if (!got && give_up(myws, call, tm_, t0)) break;
        if (!got) __builtin_amdgcn_s_sleep(FIN_SLEEP);
      }
      if (EARLY && got) {
        e0 -= 1u; e1 -= 1u;
        auto lo = [](u32 v) { return __uint_as_float(v << 16); };
        auto hi = [](u32 v) { return __uint_as_float(v & 0xffff0000u); };
        u32x4 o = {q4(lo(e0[0]), hi(e0[0]), lo(e0[1]), hi(e0[1]), inv), q4(lo(e0[2]), hi(e0[2]), lo(e0[3]), hi(e0[3]), inv),
                   q4(lo(e1[0]), hi(e1[0]), lo(e1[1]), hi(e1[1]), inv), q4(lo(e1[2]), hi(e1[2]), lo(e1[3]), hi(e1[3]), inv)};
        st128<AUX_SYS>(o, r_myout, in_off);
        st128<AUX_SYS>(u32x4{0, 0, 0, 0}, r_stg_in, stg_in_off);
        st128<AUX_SYS>(u32x4{0, 0, 0, 0}, r_stg_in, stg_in_off + 1024);
      }
      STAMP(8);
    }
  } else if (dest == rank) {
    const int grow = rank * MS + m0 + (rrow >> 5) * 64 + kh * 32 + (rrow & 31);
    st128<AUX_SYS>(u32x4{0x80808080u, 0x80808080u, 0x80808080u, 0x80808080u}, r_myout, grow * NSH + (tn & 3) * BN + (lane & 7) * 16);
  }
#endif
}

extern "C" void kernel_launch(int rank, int nranks, ncclComm_t comm, hipStream_t stream, void* const* d_in, const int* in_sizes, int n_in,
                              void* d_out, int out_size, const int* in_dtypes, int out_dtype, void* const* d_out_all,
                              void* const* d_ws_all, size_t ws_bytes) {
  if (nranks != NR || (unsigned)rank >= (unsigned)NR || ws_bytes < WS_NEED || n_in < 2 || in_sizes[0] != MS * K || in_sizes[1] != K * N || out_size != N * NSH) {
    fprintf(stderr, "unexpected problem shape\n");
    return;
  }
  Params P;
  P.x = (const u16*)d_in[0];
  P.w = (const u16*)d_in[1];
  for (int p = 0; p < NR; ++p) { P.out[p] = (char*)d_out_all[p]; P.ws[p] = (char*)d_ws_all[p]; }
  P.myws = P.ws[rank]; P.myout = (char*)d_out;
  P.rank = rank;
  hipError_t e = hipFuncSetAttribute((const void*)fused_gemm_a2a_q8, hipFuncAttributeMaxDynamicSharedMemorySize, LDS_BYTES);
  if (e != hipSuccess) { fprintf(stderr, "hipFuncSetAttribute: %s\n", hipGetErrorString(e)); return; }
  hipLaunchKernelGGL(fused_gemm_a2a_q8, dim3(NWG), dim3(512), LDS_BYTES, stream, P);
}
```

```cpp
#include <hip/hip_runtime.h>
#include <rccl/rccl.h>
#include <cstdio>

typedef unsigned short u16;
typedef unsigned int u32;
typedef unsigned long long u64;
typedef __attribute__((ext_vector_type(8))) short s16x8;
typedef __attribute__((ext_vector_type(4))) short s16x4;
typedef __attribute__((ext_vector_type(8))) __bf16 bf16x8;
typedef __attribute__((ext_vector_type(2))) __bf16 bf16x2;
typedef __attribute__((ext_vector_type(4))) float f32x4;
typedef __attribute__((ext_vector_type(2))) float f32x2;
typedef __attribute__((ext_vector_type(2))) _Float16 f16x2;
typedef __attribute__((ext_vector_type(4))) unsigned int u32x4;
#define LDS __attribute__((address_space(3)))
#define GAS __attribute__((address_space(1)))

namespace {
constexpr int EARLY = 1;
constexpr int FIN_SLEEP = 2;
constexpr int PACE = 160;
constexpr int TAILD = 2;
constexpr int PRE_SLEEP = 4;
constexpr int NR = 8;
constexpr int MS = 512, N = 4096, K = 4096;
constexpr int NSH = N / NR;
constexpr int BM = 128, BN = 128, BK = 64, KSPLIT = 2, NS = 4;
constexpr int WBYTES = BK * BN * 2, XBYTES = BM * BK * 2, STAGE = WBYTES + XBYTES;
constexpr int T = K / KSPLIT / BK;
constexpr int TD = TAILD;
constexpr int TF = T - TD, S_TOT = T + TD;
constexpr int NWG = 256;
constexpr int LDS_BYTES = NS * STAGE;
constexpr int LPW = 8;
constexpr size_t WS_CALL = 0;
constexpr size_t WS_TMO = 128;
constexpr size_t WS_L1G = 4096;
constexpr int REP = 8;
constexpr size_t WS_GG = WS_L1G + 2 * 8 * 128 * 8;
constexpr size_t WS_DBG = 512 * 1024;
constexpr size_t WS_SLAB = 1 << 20;
constexpr size_t WS_STG = WS_SLAB + (size_t)NWG * 32768;
constexpr size_t STG_BYTES = (size_t)N * NSH * 2;
constexpr size_t WS_NEED = WS_STG + 2 * STG_BYTES;
constexpr u64 DEADLINE = 100000000ull;
constexpr int AUX_SC1 = 16, AUX_SYS = 17;

struct Params {
  const u16* x; const u16* w;
  char* myws; char* myout;
  char* out[NR]; char* ws[NR];
  int rank;
};

template <int N_> __device__ __forceinline__ void wait_vm() { asm volatile("s_waitcnt vmcnt(%0)" ::"i"(N_) : "memory"); }
__device__ __forceinline__ void bar() { asm volatile("s_barrier" ::: "memory"); }
__device__ __forceinline__ void bar_lgkm() {
  __builtin_amdgcn_sched_barrier(0);
  __builtin_amdgcn_s_waitcnt(0xC07F);
  asm volatile("s_barrier" ::: "memory");
  __builtin_amdgcn_sched_barrier(0);
}
__device__ __forceinline__ u32 ld_agent32(const char* p) { return __hip_atomic_load((const GAS u32*)p, __ATOMIC_RELAXED, __HIP_MEMORY_SCOPE_AGENT); }
__device__ __forceinline__ u64 ld_sys64(const char* p) { return __hip_atomic_load((const GAS u64*)p, __ATOMIC_RELAXED, __HIP_MEMORY_SCOPE_SYSTEM); }
__device__ __forceinline__ void st_agent32(char* p, u32 v) { __hip_atomic_store((GAS u32*)p, v, __ATOMIC_RELAXED, __HIP_MEMORY_SCOPE_AGENT); }
__device__ __forceinline__ void st_agent64(char* p, u64 v) { __hip_atomic_store((GAS u64*)p, v, __ATOMIC_RELAXED, __HIP_MEMORY_SCOPE_AGENT); }
__device__ __forceinline__ void st_sys64(char* p, u64 v) { __hip_atomic_store((GAS u64*)p, v, __ATOMIC_RELAXED, __HIP_MEMORY_SCOPE_SYSTEM); }
__device__ __forceinline__ u32 lds_ld(const char LDS* p) { return __hip_atomic_load((const LDS u32*)p, __ATOMIC_RELAXED, __HIP_MEMORY_SCOPE_WORKGROUP); }
__device__ __forceinline__ void lds_st(char LDS* p, u32 v) { __hip_atomic_store((LDS u32*)p, v, __ATOMIC_RELAXED, __HIP_MEMORY_SCOPE_WORKGROUP); }

__device__ __forceinline__ bool give_up(char* myws, u32 call, u32 tmo, u64 t0) {
  bool bad = tmo == call || (__builtin_amdgcn_s_memrealtime() - t0) > DEADLINE;
  bad = __builtin_amdgcn_readfirstlane(bad);
  if (bad) st_agent32(myws + WS_TMO, call);
  return bad;
}
__device__ __forceinline__ float wave_max(float v) {
  int x = __float_as_int(v);
  x = max(x, __builtin_amdgcn_update_dpp(0, x, 0x111, 0xf, 0xf, false));
  x = max(x, __builtin_amdgcn_update_dpp(0, x, 0x112, 0xf, 0xf, false));
  x = max(x, __builtin_amdgcn_update_dpp(0, x, 0x114, 0xf, 0xf, false));
  x = max(x, __builtin_amdgcn_update_dpp(0, x, 0x118, 0xf, 0xf, false));
  int a = __builtin_amdgcn_readlane(x, 15), b = __builtin_amdgcn_readlane(x, 31), c = __builtin_amdgcn_readlane(x, 47), d = __builtin_amdgcn_readlane(x, 63);
  return __int_as_float(max(max(a, b), max(c, d)));
}
template <int AUX, typename R> __device__ __forceinline__ void st128(u32x4 v, R rsrc, int voff) { __builtin_amdgcn_raw_buffer_store_b128(v, rsrc, voff, 0, AUX); }
__device__ __forceinline__ u32 enc(u32 v) { return __builtin_elementwise_add_sat(v, 1u); }
__device__ __forceinline__ bool nz4(u32x4 v) { return v[0] != 0 && v[1] != 0 && v[2] != 0 && v[3] != 0; }
__device__ __forceinline__ bool has80(u32 v) { u32 t = v ^ 0x80808080u; return ((t - 0x01010101u) & ~t & 0x80808080u) != 0; }
__device__ __forceinline__ u32 q4(float a, float b, float c, float d, float inv) {
  const float M = 12582912.0f;
  u32 qa = __float_as_uint(__builtin_fmaf(a, inv, M)), qb = __float_as_uint(__builtin_fmaf(b, inv, M));
  u32 qc = __float_as_uint(__builtin_fmaf(c, inv, M)), qd = __float_as_uint(__builtin_fmaf(d, inv, M));
  u32 lo = __builtin_amdgcn_perm(qb, qa, 0x0c0c0400u), hi = __builtin_amdgcn_perm(qd, qc, 0x04000c0cu);
  return lo | hi;
}
__device__ __forceinline__ u32 pack_bf16(float a, float b) {
  bf16x2 v = __builtin_convertvector(f32x2{a, b}, bf16x2);
  return __builtin_bit_cast(u32, v);
}
#define STAMP(i) do {} while (0)
constexpr int L_BASE = ((S_TOT - 4) % NS) * STAGE;
static_assert(((S_TOT - 3) % NS) == ((S_TOT - 4) % NS) + 1, "two adjacent dead slots");
static_assert(NS == 4 && (TD == 0 || (TD >= 2 && TD % 2 == 0)), "tail code");
constexpr int L_SCALE = L_BASE, L_STATE = L_BASE + 4  , L_TILE = L_BASE + 1024, TROW = 528;
}

__global__ __launch_bounds__(512, 2) void fused_gemm_a2a_q8(Params P) {
#if __HIP_DEVICE_COMPILE__
  extern __shared__ __attribute__((aligned(16))) char smem_[];
  char LDS* const smem = (char LDS*)smem_;
  const int tid = threadIdx.x, lane = tid & 63;
  const int wid = __builtin_amdgcn_readfirstlane(tid >> 6);
  const int b = blockIdx.x, grp = b & 7, idx = b >> 3;
  const int kh = grp & 1, tn = (grp >> 1) * 8 + (idx & 7), tm = idx >> 3;
  const int partner = b ^ 1;
  const int n0 = tn * BN, m0 = tm * BM, k0 = kh * (K / KSPLIT);
  const int dest = tn >> 2;
  const u16* xp = P.x; const u16* wp = P.w;
  char* myws = P.myws; char* myout = P.myout; int rank = P.rank;
  char* out_dest = P.out[dest]; char* ws_dest = P.ws[dest];
  asm volatile("" : "+s"(xp), "+s"(wp), "+s"(myws), "+s"(myout), "+s"(rank), "+s"(out_dest), "+s"(ws_dest));
  STAMP(0);
  const int rrow = wid * 8 + (lane >> 3);
  const int in_off = (grp * MS + ((idx >> 2) & 3) * BM + (rrow >> 5) * 64 + (idx >> 4) * 32 + (rrow & 31)) * NSH + (idx & 3) * BN + (lane & 7) * 16;
  auto r_myout = __builtin_amdgcn_make_buffer_rsrc((void*)myout, 0, N * NSH, 0x00020000);
  const bool recv = grp != rank;
  u32 call;
  char* ws_lane_all = nullptr;
  if (wid >= 4) {
    const int lw = wid - 4;
    u32 callv = ld_agent32(myws + WS_CALL);
    char* ws_lane = P.ws[lane & 7];
    auto rw = __builtin_amdgcn_make_buffer_rsrc((void*)wp, 0, K * N * 2, 0x00020000);
    auto rx = __builtin_amdgcn_make_buffer_rsrc((void*)xp, 0, MS * K * 2, 0x00020000);
    int vw[4], vx[4];
#pragma unroll
    for (int j = 0; j < 4; ++j) {
      int k = 16 * lw + 4 * j + (lane >> 4);
      int F = (k & 3) | ((k >> 1) & 4);
      vw[j] = k * (N * 2) + (((lane & 15) ^ (F << 1)) * 16) + n0 * 2;
      int r = (lw * 4 + j) * 8 + (lane >> 3);
      vx[j] = (m0 + r) * (K * 2) + (((lane & 7) ^ ((r >> 1) & 7)) * 16);
    }
    const int rj = (tm + tn) & 3;
    auto issue = [&](int s, int slot_off) {
      const int ss = (s + ((s & 3) == rj ? 4 : 0) + grp * 4) & (T - 1);
      const int sw = (k0 + ss * BK) * (N * 2), sx = (k0 + ss * BK) * 2;
#pragma unroll
      for (int j = 0; j < 4; ++j) __builtin_amdgcn_raw_ptr_buffer_load_lds(rw, (LDS void*)(smem + slot_off + (lw * 4 + j) * 1024), 16, vw[j], sw, 0, 0);
#pragma unroll
      for (int j = 0; j < 4; ++j) __builtin_amdgcn_raw_ptr_buffer_load_lds(rx, (LDS void*)(smem + slot_off + WBYTES + (lw * 4 + j) * 1024), 16, vx[j], sx, 0, 0);
    };
    int vxk[2], lxk[2];
#pragma unroll
    for (int j = 0; j < 2; ++j) {
      const int ik = (lw >> 1) * 8 + kh * 4 + (lw & 1) * 2 + j;
      const int r = ik * 8 + (lane >> 3);
      vxk[j] = (m0 + r) * (K * 2) + (((lane & 7) ^ ((r >> 1) & 7)) * 16);
      lxk[j] = WBYTES + ik * 1024;
    }
    auto issue_half = [&](int h, int slot_off) {
      const bool own = h < TD;
      const int s = TF + (own ? h : h - TD);
      const int ss = (s + ((s & 3) == rj ? 4 : 0) + (own ? grp : (partner & 7)) * 4) & (T - 1);
      const int kk = (own ? k0 : K / KSPLIT - k0) + ss * BK;
      const int sw = kk * (N * 2), sx = kk * 2;
#pragma unroll
      for (int j = 0; j < 4; ++j) __builtin_amdgcn_raw_ptr_buffer_load_lds(rw, (LDS void*)(smem + slot_off + (lw * 4 + j) * 1024), 16, vw[j], sw, 0, 0);
#pragma unroll
      for (int j = 0; j < 2; ++j) __builtin_amdgcn_raw_ptr_buffer_load_lds(rx, (LDS void*)(smem + slot_off + lxk[j]), 16, vxk[j], sx, 0, 0);
    };
#pragma unroll
    for (int s = 0; s < NS - 1; ++s) issue(s, s * STAGE);
    int slot = (NS - 1) * STAGE;
    auto next = [&]() { slot += STAGE; if (slot == NS * STAGE) slot = 0; };
#pragma unroll 1
    for (int s = 0; s + 3 < TF; ++s) { wait_vm<16>(); bar(); issue(s + 3, slot); next(); }
    if (TD == 0) {
      wait_vm<16>(); bar(); wait_vm<8>(); bar(); wait_vm<0>(); bar();
    } else {
      wait_vm<16>(); bar(); issue_half(0, slot); next();
      wait_vm<14>(); bar(); issue_half(1, slot); next();
#pragma unroll 1
      for (int h = 2; h < 2 * TD; ++h) { wait_vm<12>(); bar(); issue_half(h, slot); next(); }
      wait_vm<12>(); bar(); wait_vm<6>(); bar(); wait_vm<0>(); bar();
    }
    call = __builtin_amdgcn_readfirstlane(callv) + 1;
    ws_lane_all = ws_lane;
    if (wid == 4 && lane == 0) lds_st(smem + L_STATE, 0);
  } else {
    u32 callv = ld_agent32(myws + WS_CALL);
    if (recv) {
      st128<AUX_SYS>(u32x4{0x80808080u, 0x80808080u, 0x80808080u, 0x80808080u}, r_myout, in_off);
      st128<AUX_SYS>(u32x4{0x80808080u, 0x80808080u, 0x80808080u, 0x80808080u}, r_myout, in_off + 64 * NSH);
    }
    f32x4 acc[4][4];
    const int wm = wid & 1, wn = wid >> 1;
    const int g = lane >> 4, q = (lane & 15) >> 2, p = lane & 3;
    const int F = q | ((g & 1) << 2);
    int aw[4], ax[2][2];
#pragma unroll
    for (int c = 0; c < 4; ++c) aw[c] = (8 * g + q) * 256 + ((((4 * wn + c) ^ F) << 1) | (p >> 1)) * 16 + 8 * (p & 1);
#pragma unroll
    for (int ks = 0; ks < 2; ++ks) {
      int r = wm * 64 + (lane & 15), a = WBYTES + r * 128 + (((4 * ks + g) ^ ((r >> 1) & 7)) * 16);
      ax[ks][0] = a + kh * 4096; ax[ks][1] = a + (1 - kh) * 4096;
    }
#pragma unroll
    for (int i = 0; i < 4; ++i)
#pragma unroll
      for (int j = 0; j < 4; ++j) acc[i][j] = f32x4{0, 0, 0, 0};
    s16x8 W0[4][2], X0[4][2], W1[4][2], X1[4][2];
    auto rd = [&](s16x8(&W)[4][2], s16x8(&X)[4][2], int so) {
#pragma unroll
      for (int ks = 0; ks < 2; ++ks) {
#pragma unroll
        for (int j = 0; j < 4; ++j) X[j][ks] = *(LDS s16x8*)(smem + so + ax[ks][j >> 1] + (j & 1) * 2048);
#pragma unroll
        for (int c = 0; c < 4; ++c) {
          s16x4 lo = __builtin_amdgcn_ds_read_tr16_b64_v4i16((LDS s16x4*)(smem + so + aw[c] + (32 * ks) * 256));
          s16x4 hi = __builtin_amdgcn_ds_read_tr16_b64_v4i16((LDS s16x4*)(smem + so + aw[c] + (32 * ks + 4) * 256));
          W[c][ks] = __builtin_shufflevector(lo, hi, 0, 1, 2, 3, 4, 5, 6, 7);
        }
      }
    };
    auto mm = [&](s16x8(&W)[4][2], s16x8(&X)[4][2]) {
#pragma unroll
      for (int ks = 0; ks < 2; ++ks)
#pragma unroll
        for (int c = 0; c < 4; ++c)
#pragma unroll
          for (int jj = 0; jj < 4; ++jj) {
            const int j = (c & 1) ? 3 - jj : jj;
            acc[c][j] = __builtin_amdgcn_mfma_f32_16x16x32_bf16(__builtin_bit_cast(bf16x8, W[c][ks]), __builtin_bit_cast(bf16x8, X[j][ks]), acc[c][j], 0, 0, 0);
          }
    };
    auto sched = [&]() {
#pragma unroll
      for (int i = 0; i < 32; ++i) {
        __builtin_amdgcn_sched_group_barrier(0x8, 1, 0);
        if (i < 24) __builtin_amdgcn_sched_group_barrier(0x100, 1, 0);
      }
    };
    auto rdh = [&](s16x8(&W)[4][2], s16x8(&X)[4][2], int so) {
#pragma unroll
      for (int ks = 0; ks < 2; ++ks) {
#pragma unroll
        for (int j = 0; j < 2; ++j) X[j][ks] = *(LDS s16x8*)(smem + so + ax[ks][0] + j * 2048);
#pragma unroll
        for (int c = 0; c < 4; ++c) {
          s16x4 lo = __builtin_amdgcn_ds_read_tr16_b64_v4i16((LDS s16x4*)(smem + so + aw[c] + (32 * ks) * 256));
          s16x4 hi = __builtin_amdgcn_ds_read_tr16_b64_v4i16((LDS s16x4*)(smem + so + aw[c] + (32 * ks + 4) * 256));
          W[c][ks] = __builtin_shufflevector(lo, hi, 0, 1, 2, 3, 4, 5, 6, 7);
        }
      }
    };
    auto mmj = [&](s16x8(&W)[4][2], s16x8(&X)[4][2], int j0) {
#pragma unroll
      for (int ks = 0; ks < 2; ++ks)
#pragma unroll
        for (int c = 0; c < 4; ++c)
#pragma unroll
          for (int j = j0; j < j0 + 2; ++j)
            acc[c][j] = __builtin_amdgcn_mfma_f32_16x16x32_bf16(__builtin_bit_cast(bf16x8, W[c][ks]), __builtin_bit_cast(bf16x8, X[j][ks]), acc[c][j], 0, 0, 0);
    };
    auto schedh = [&]() {
#pragma unroll
      for (int i = 0; i < 16; ++i) {
        __builtin_amdgcn_sched_group_barrier(0x8, 1, 0);
        if (i < 10) __builtin_amdgcn_sched_group_barrier(0x100, 2, 0);
      }
    };
    auto rs = __builtin_amdgcn_make_buffer_rsrc((void*)(myws + WS_SLAB), 0, NWG * 32768, 0x00020000);
    const int so_out = (partner * 4 + wid) * 8192, so_in = (b * 4 + wid) * 8192;
    u32 tag;
    constexpr int NX = 4;
    u32 in_sc;
    auto send_partial = [&]() {
      call = __builtin_amdgcn_readfirstlane(callv) + 1;
      tag = call & 1;
      float pm = 0.f;
#pragma unroll
      for (int c = 0; c < 4; ++c)
#pragma unroll
        for (int j = 2; j < 4; ++j)
#pragma unroll
          for (int r = 0; r < 4; ++r) pm = fmaxf(pm, fabsf(acc[c][j][r]));
      u32 eb = __float_as_uint(pm) & 0x7f800000u;
      eb = min(max(eb, 15u << 23), 254u << 23);
      const float up = __uint_as_float((268u << 23) - eb);
      const u32 dn = eb - (14u << 23);
#pragma unroll
      for (int c = 0; c < 4; ++c) {
        u32x4 e;
#pragma unroll
        for (int j = 0; j < 2; ++j)
#pragma unroll
          for (int r = 0; r < 2; ++r) {
            f16x2 h = __builtin_convertvector(f32x2{acc[c][2 + j][2 * r] * up, acc[c][2 + j][2 * r + 1] * up}, f16x2);
            e[j * 2 + r] = (__builtin_bit_cast(u32, h) & ~1u) | tag;
          }
        st128<AUX_SC1>(e, rs, lane * 16 + c * 1024 + so_out);
      }
      __builtin_amdgcn_raw_buffer_store_b32(dn | tag, rs, lane * 4 + 4096, so_out, AUX_SC1);
      STAMP(2);
    };
    u32x4 in[NX];
    auto load_partial = [&]() {
#pragma unroll
      for (int i = 0; i < NX; ++i) in[i] = __builtin_amdgcn_raw_buffer_load_b128(rs, lane * 16 + i * 1024, so_in, AUX_SC1);
      in_sc = __builtin_amdgcn_raw_buffer_load_b32(rs, lane * 4 + 4096, so_in, AUX_SC1);
    };
    int slot = 0;
    auto next = [&]() { slot += STAGE; if (slot == NS * STAGE) slot = 0; };
    bar();
    rd(W0, X0, slot); next();
    for (int s = 1; s < TF - 1; s += 2) {
      bar_lgkm();
      rd(W1, X1, slot); mm(W0, X0); sched(); next();
      bar_lgkm();
      rd(W0, X0, slot); mm(W1, X1); sched(); next();
    }
    bar_lgkm();
    rd(W1, X1, slot); mm(W0, X0); sched(); next();
    if (TD == 0) {
      mmj(W1, X1, 2); send_partial(); mmj(W1, X1, 0);
      load_partial();
    } else {
      bar_lgkm();
      rdh(W0, X0, slot); mmj(W1, X1, 2); schedh(); next();
      send_partial();
      mmj(W1, X1, 0);
#pragma unroll 1
      for (int h = 1; h < 2 * TD - 3; h += 2) {
        bar_lgkm();
        rdh(W1, X1, slot); mmj(W0, X0, 0); schedh(); next();
        bar_lgkm();
        rdh(W0, X0, slot); mmj(W1, X1, 0); schedh(); next();
      }
      bar_lgkm();
      rdh(W1, X1, slot); mmj(W0, X0, 0); schedh(); next();
      bar_lgkm();
      load_partial();
      rdh(W0, X0, slot); mmj(W1, X1, 0); schedh(); next();
      bar_lgkm();
      rdh(W1, X1, slot); mmj(W0, X0, 0); schedh();
      mmj(W1, X1, 0);
    }
    STAMP(1);
    bool ok = true;
    {
      u64 t0 = __builtin_amdgcn_s_memrealtime();
      u32 tm_ = 0;
      for (;;) {
        u32 an = in_sc, orr = in_sc;
#pragma unroll
        for (int i = 0; i < NX; ++i) { an &= (in[i][0] & in[i][1]) & (in[i][2] & in[i][3]); orr |= (in[i][0] | in[i][1]) | (in[i][2] | in[i][3]); }
        if (__all(((tag ? an : ~orr) & 1u) != 0)) break;
        if (give_up(myws, call, tm_, t0)) { ok = false; break; }
        asm volatile("" ::: "memory");
        load_partial();
        tm_ = ld_agent32(myws + WS_TMO);
      }
    }
    STAMP(3);
    float am = 0.f;
#pragma unroll
    for (int c = 0; c < 4; ++c)
#pragma unroll
      for (int j = 0; j < 2; ++j) {
        const float dn = __uint_as_float(in_sc & ~1u);
#pragma unroll
        for (int r = 0; r < 2; ++r) {
          float a0 = acc[c][j][2 * r], a1 = acc[c][j][2 * r + 1];
          asm("v_fma_mix_f32 %0, %1, %2, %0 op_sel_hi:[1,0,0]" : "+v"(a0) : "v"(in[c][j * 2 + r]), "v"(dn));
          asm("v_fma_mix_f32 %0, %1, %2, %0 op_sel:[1,0,0] op_sel_hi:[1,0,0]" : "+v"(a1) : "v"(in[c][j * 2 + r]), "v"(dn));
          acc[c][j][2 * r] = a0; acc[c][j][2 * r + 1] = a1;
        }
#pragma unroll
        for (int r = 0; r < 4; ++r) am = fmaxf(am, fabsf(acc[c][j][r]));
      }
    am = wave_max(am);
    if (lane == 0 && ok) {
      const u64 gr = ((u64)call << 32) | __float_as_uint(am);
      st_agent64(myws + WS_L1G + (size_t)((call & 1) * 8 + grp) * 1024 + (idx * 4 + wid) * 8, gr);
    }
#pragma unroll
    for (int c = 0; c < 4; ++c)
#pragma unroll
      for (int j = 0; j < 2; ++j) {
        int row = wm * 32 + j * 16 + (lane & 15), col = wn * 64 + c * 16 + 4 * g;
        *(LDS f32x4*)(smem + L_TILE + row * TROW + col * 4) = acc[c][j];
      }
    STAMP(4);
  }
  bar_lgkm();
  const int par = call & 1;
  f32x4 yv[4];
#pragma unroll
  for (int i = 0; i < 4; ++i) yv[i] = *(LDS f32x4*)(smem + L_TILE + rrow * TROW + (lane & 7) * 64 + i * 16);
  auto r_stg_in = __builtin_amdgcn_make_buffer_rsrc((void*)(myws + WS_STG + par * STG_BYTES), 0, (int)STG_BYTES, 0x00020000);
  const int stg_in_off = ((grp * 32 + idx) * 8 + wid) * 2048 + lane * 16;
  const int stg_probe = ((grp * 32 + idx) * 8 + wid) * 2048 + 2044;
  bool sent = false, got = false, rearmed = false;
  u32x4 e0, e1;
  u64 t0 = __builtin_amdgcn_s_memrealtime();
  const bool can_early = EARLY && dest != rank;
  auto send_early = [&]() {
    auto r_stg = __builtin_amdgcn_make_buffer_rsrc((void*)(ws_dest + WS_STG + par * STG_BYTES), 0, (int)STG_BYTES, 0x00020000);
    const int so = ((rank * 32 + ((tn & 3) + 4 * tm + 16 * kh)) * 8 + wid) * 2048;
    u32x4 a = {enc(pack_bf16(yv[0][0], yv[0][1])), enc(pack_bf16(yv[0][2], yv[0][3])), enc(pack_bf16(yv[1][0], yv[1][1])), enc(pack_bf16(yv[1][2], yv[1][3]))};
    u32x4 c = {enc(pack_bf16(yv[2][0], yv[2][1])), enc(pack_bf16(yv[2][2], yv[2][3])), enc(pack_bf16(yv[3][0], yv[3][1])), enc(pack_bf16(yv[3][2], yv[3][3]))};
    st128<AUX_SYS>(a, r_stg, lane * 16 + so);
    st128<AUX_SYS>(c, r_stg, lane * 16 + 1024 + so);
    sent = true;
    STAMP(9);
  };
  if (wid == 4) {
    const char* gsrc = myws + WS_GG + (size_t)((par * REP + (idx & (REP - 1))) * 64 + lane) * 8;
    u64 v; bool ok = true;
    for (;;) {
      v = ld_sys64(gsrc);
      u32 tm_ = ld_agent32(myws + WS_TMO);
      if (__all((u32)(v >> 32) == call)) { ok = tm_ != call; break; }
      if (give_up(myws, call, tm_, t0)) { ok = false; break; }
    }
    float gm = wave_max(__uint_as_float((u32)v));
    if (lane == 0) {
      lds_st(smem + L_SCALE, __float_as_uint(gm > 0.f ? 127.0f / gm : 0.f));
      __builtin_amdgcn_s_waitcnt(0xC07F);
      lds_st(smem + L_STATE, ok ? 1u : 2u);
      if (b == 0) st_agent32(myws + WS_CALL, call);
    }
    STAMP(6);
  } else {
    if (wid == 5 && idx == 0) {
      const char* l1 = myws + WS_L1G + (size_t)(par * 8 + grp) * 1024 + lane * 16;
      char* dst = ws_lane_all + WS_GG + (size_t)((par * REP + (lane >> 3)) * 64 + rank * 8 + grp) * 8;
      auto rl = __builtin_amdgcn_make_buffer_rsrc((void*)myws, 0, 1 << 20, 0x00020000);
      for (;;) {
        asm volatile("" ::: "memory");
        u32x4 v = __builtin_amdgcn_raw_buffer_load_b128(rl, (int)(l1 - myws), 0, AUX_SC1);
        u32 tm_ = ld_agent32(myws + WS_TMO);
        if (__all(v[1] == call && v[3] == call)) {
          float gm = wave_max(fmaxf(__uint_as_float(v[0]), __uint_as_float(v[2])));
          st_sys64(dst, ((u64)call << 32) | __float_as_uint(gm));
          break;
        }
        if (give_up(myws, call, tm_, t0)) break;
      }
      STAMP(5);
    }
    const u64 due = t0 + (u64)(wid - (wid > 4)) * PACE;
    while (__builtin_amdgcn_readfirstlane(lds_ld(smem + L_STATE)) == 0) {
      if (can_early && !sent && __builtin_amdgcn_s_memrealtime() >= due) {
        send_early();
      } else if (EARLY && recv && !got && (sent || !can_early)) {
        asm volatile("" ::: "memory");
        if (__builtin_amdgcn_readfirstlane(__builtin_amdgcn_raw_buffer_load_b32(r_stg_in, stg_probe, 0, AUX_SYS)) != 0) {
          e0 = __builtin_amdgcn_raw_buffer_load_b128(r_stg_in, stg_in_off, 0, AUX_SYS);
          e1 = __builtin_amdgcn_raw_buffer_load_b128(r_stg_in, stg_in_off + 1024, 0, AUX_SYS);
          got = __all(nz4(e0) && nz4(e1));
          if (got) {
            rearmed = true;
            st128<AUX_SYS>(u32x4{0, 0, 0, 0}, r_stg_in, stg_in_off);
            st128<AUX_SYS>(u32x4{0, 0, 0, 0}, r_stg_in, stg_in_off + 1024);
          }
        } else __builtin_amdgcn_s_sleep(PRE_SLEEP);
      } else {
        __builtin_amdgcn_s_sleep(1);
      }
    }
  }
  __builtin_amdgcn_s_waitcnt(0xC07F);
  const u32 state = __builtin_amdgcn_readfirstlane(lds_ld(smem + L_STATE));
  const float inv = __uint_as_float(lds_ld(smem + L_SCALE));
  if (state == 1) {
    if (!sent) {
      u32x4 o = {q4(yv[0][0], yv[0][1], yv[0][2], yv[0][3], inv), q4(yv[1][0], yv[1][1], yv[1][2], yv[1][3], inv),
                 q4(yv[2][0], yv[2][1], yv[2][2], yv[2][3], inv), q4(yv[3][0], yv[3][1], yv[3][2], yv[3][3], inv)};
      const int grow = rank * MS + m0 + (rrow >> 5) * 64 + kh * 32 + (rrow & 31);
      auto ro = __builtin_amdgcn_make_buffer_rsrc((void*)out_dest, 0, N * NSH, 0x00020000);
      st128<AUX_SYS>(o, ro, grow * NSH + (tn & 3) * BN + (lane & 7) * 16);
    }
    STAMP(7);
    if (recv) {
      while (!got) {
        asm volatile("" ::: "memory");
        u32x4 o = __builtin_amdgcn_raw_buffer_load_b128(r_myout, in_off, 0, AUX_SYS);
        if (EARLY) {
          e0 = __builtin_amdgcn_raw_buffer_load_b128(r_stg_in, stg_in_off, 0, AUX_SYS);
          e1 = __builtin_amdgcn_raw_buffer_load_b128(r_stg_in, stg_in_off + 1024, 0, AUX_SYS);
        }
        u32 tm_ = ld_agent32(myws + WS_TMO);
        if (__all(!(has80(o[0]) || has80(o[1]) || has80(o[2]) || has80(o[3])))) break;
        if (EARLY) got = __all(nz4(e0) && nz4(e1));
        if (!got && give_up(myws, call, tm_, t0)) break;
        if (!got) __builtin_amdgcn_s_sleep(FIN_SLEEP);
      }
      if (EARLY && got) {
        e0 -= 1u; e1 -= 1u;
        auto lo = [](u32 v) { return __uint_as_float(v << 16); };
        auto hi = [](u32 v) { return __uint_as_float(v & 0xffff0000u); };
        u32x4 o = {q4(lo(e0[0]), hi(e0[0]), lo(e0[1]), hi(e0[1]), inv), q4(lo(e0[2]), hi(e0[2]), lo(e0[3]), hi(e0[3]), inv),
                   q4(lo(e1[0]), hi(e1[0]), lo(e1[1]), hi(e1[1]), inv), q4(lo(e1[2]), hi(e1[2]), lo(e1[3]), hi(e1[3]), inv)};
        st128<AUX_SYS>(o, r_myout, in_off);
        if (!rearmed) {
          st128<AUX_SYS>(u32x4{0, 0, 0, 0}, r_stg_in, stg_in_off);
          st128<AUX_SYS>(u32x4{0, 0, 0, 0}, r_stg_in, stg_in_off + 1024);
        }
      }
      STAMP(8);
    }
  } else if (dest == rank) {
    const int grow = rank * MS + m0 + (rrow >> 5) * 64 + kh * 32 + (rrow & 31);
    st128<AUX_SYS>(u32x4{0x80808080u, 0x80808080u, 0x80808080u, 0x80808080u}, r_myout, grow * NSH + (tn & 3) * BN + (lane & 7) * 16);
  }
#endif
}

extern "C" void kernel_launch(int rank, int nranks, ncclComm_t comm, hipStream_t stream, void* const* d_in, const int* in_sizes, int n_in,
                              void* d_out, int out_size, const int* in_dtypes, int out_dtype, void* const* d_out_all,
                              void* const* d_ws_all, size_t ws_bytes) {
  if (nranks != NR || (unsigned)rank >= (unsigned)NR || ws_bytes < WS_NEED || n_in < 2 || in_sizes[0] != MS * K || in_sizes[1] != K * N || out_size != N * NSH) {
    fprintf(stderr, "unexpected problem shape\n");
    return;
  }
  Params P;
  P.x = (const u16*)d_in[0];
  P.w = (const u16*)d_in[1];
  for (int p = 0; p < NR; ++p) { P.out[p] = (char*)d_out_all[p]; P.ws[p] = (char*)d_ws_all[p]; }
  P.myws = P.ws[rank]; P.myout = (char*)d_out;
  P.rank = rank;
  hipError_t e = hipFuncSetAttribute((const void*)fused_gemm_a2a_q8, hipFuncAttributeMaxDynamicSharedMemorySize, LDS_BYTES);
  if (e != hipSuccess) { fprintf(stderr, "hipFuncSetAttribute: %s\n", hipGetErrorString(e)); return; }
  hipLaunchKernelGGL(fused_gemm_a2a_q8, dim3(NWG), dim3(512), LDS_BYTES, stream, P);
}
```

```cpp
#include <hip/hip_runtime.h>
#include <rccl/rccl.h>
#include <cstdio>

typedef unsigned short u16;
typedef unsigned int u32;
typedef unsigned long long u64;
typedef __attribute__((ext_vector_type(8))) short s16x8;
typedef __attribute__((ext_vector_type(4))) short s16x4;
typedef __attribute__((ext_vector_type(8))) __bf16 bf16x8;
typedef __attribute__((ext_vector_type(2))) __bf16 bf16x2;
typedef __attribute__((ext_vector_type(4))) float f32x4;
typedef __attribute__((ext_vector_type(2))) float f32x2;
typedef __attribute__((ext_vector_type(2))) _Float16 f16x2;
typedef __attribute__((ext_vector_type(4))) unsigned int u32x4;
#define LDS __attribute__((address_space(3)))
#define GAS __attribute__((address_space(1)))

namespace {
constexpr int EARLY = 1;
constexpr int PACE = 160;
constexpr int TAILD = 2;
constexpr int SPEC = 40;
constexpr int SPACE = 70;
constexpr int PRE_SLEEP = 4;
constexpr int NR = 8;
constexpr int MS = 512, N = 4096, K = 4096;
constexpr int NSH = N / NR;
constexpr int BM = 128, BN = 128, BK = 64, KSPLIT = 2, NS = 4;
constexpr int WBYTES = BK * BN * 2, XBYTES = BM * BK * 2, STAGE = WBYTES + XBYTES;
constexpr int T = K / KSPLIT / BK;
constexpr int TD = TAILD;
constexpr int TF = T - TD, S_TOT = T + TD;
constexpr int NWG = 256;
constexpr int LDS_BYTES = NS * STAGE;
constexpr int LPW = 8;
constexpr size_t WS_CALL = 0;
constexpr size_t WS_TMO = 128;
constexpr size_t WS_L1G = 4096;
constexpr int REP = 8, RUSE = 4;
constexpr size_t WS_GG = WS_L1G + 2 * 8 * 128 * 8;
constexpr size_t WS_DBG = 512 * 1024;
constexpr size_t WS_SLAB = 1 << 20;
constexpr size_t WS_STG = WS_SLAB + (size_t)NWG * 32768;
constexpr size_t STG_BYTES = (size_t)N * NSH * 2;
constexpr size_t WS_SPC = WS_STG + 2 * STG_BYTES;
constexpr size_t SPC_BYTES = (size_t)N * NSH;
constexpr size_t WS_HDR = 128 * 1024;
constexpr size_t WS_NEED = WS_SPC + 2 * SPC_BYTES;
constexpr u64 DEADLINE = 100000000ull;
constexpr int AUX_SC1 = 16, AUX_SYS = 17;

struct Params {
  const u16* x; const u16* w;
  char* myws; char* myout;
  char* out[NR]; char* ws[NR];
  int rank;
};

template <int N_> __device__ __forceinline__ void wait_vm() { asm volatile("s_waitcnt vmcnt(%0)" ::"i"(N_) : "memory"); }
__device__ __forceinline__ void bar() { asm volatile("s_barrier" ::: "memory"); }
__device__ __forceinline__ void bar_lgkm() {
  __builtin_amdgcn_sched_barrier(0);
  __builtin_amdgcn_s_waitcnt(0xC07F);
  asm volatile("s_barrier" ::: "memory");
  __builtin_amdgcn_sched_barrier(0);
}
__device__ __forceinline__ u32 ld_agent32(const char* p) { return __hip_atomic_load((const GAS u32*)p, __ATOMIC_RELAXED, __HIP_MEMORY_SCOPE_AGENT); }
__device__ __forceinline__ u64 ld_sys64(const char* p) { return __hip_atomic_load((const GAS u64*)p, __ATOMIC_RELAXED, __HIP_MEMORY_SCOPE_SYSTEM); }
__device__ __forceinline__ void st_agent32(char* p, u32 v) { __hip_atomic_store((GAS u32*)p, v, __ATOMIC_RELAXED, __HIP_MEMORY_SCOPE_AGENT); }
__device__ __forceinline__ void st_agent64(char* p, u64 v) { __hip_atomic_store((GAS u64*)p, v, __ATOMIC_RELAXED, __HIP_MEMORY_SCOPE_AGENT); }
__device__ __forceinline__ void st_sys64(char* p, u64 v) { __hip_atomic_store((GAS u64*)p, v, __ATOMIC_RELAXED, __HIP_MEMORY_SCOPE_SYSTEM); }
__device__ __forceinline__ u32 lds_ld(const char LDS* p) { return __hip_atomic_load((const LDS u32*)p, __ATOMIC_RELAXED, __HIP_MEMORY_SCOPE_WORKGROUP); }
__device__ __forceinline__ void lds_st(char LDS* p, u32 v) { __hip_atomic_store((LDS u32*)p, v, __ATOMIC_RELAXED, __HIP_MEMORY_SCOPE_WORKGROUP); }

__device__ __forceinline__ bool give_up(char* myws, u32 call, u32 tmo, u64 t0) {
  bool bad = tmo == call || (__builtin_amdgcn_s_memrealtime() - t0) > DEADLINE;
  bad = __builtin_amdgcn_readfirstlane(bad);
  if (bad) st_agent32(myws + WS_TMO, call);
  return bad;
}
__device__ __forceinline__ float wave_max(float v) {
  int x = __float_as_int(v);
  x = max(x, __builtin_amdgcn_update_dpp(0, x, 0x111, 0xf, 0xf, false));
  x = max(x, __builtin_amdgcn_update_dpp(0, x, 0x112, 0xf, 0xf, false));
  x = max(x, __builtin_amdgcn_update_dpp(0, x, 0x114, 0xf, 0xf, false));
  x = max(x, __builtin_amdgcn_update_dpp(0, x, 0x118, 0xf, 0xf, false));
  int a = __builtin_amdgcn_readlane(x, 15), b = __builtin_amdgcn_readlane(x, 31), c = __builtin_amdgcn_readlane(x, 47), d = __builtin_amdgcn_readlane(x, 63);
  return __int_as_float(max(max(a, b), max(c, d)));
}
template <int AUX, typename R> __device__ __forceinline__ void st128(u32x4 v, R rsrc, int voff) { __builtin_amdgcn_raw_buffer_store_b128(v, rsrc, voff, 0, AUX); }
__device__ __forceinline__ u32 enc(u32 v) { return __builtin_elementwise_add_sat(v, 1u); }
__device__ __forceinline__ bool nz4(u32x4 v) { return v[0] != 0 && v[1] != 0 && v[2] != 0 && v[3] != 0; }
__device__ __forceinline__ bool has80(u32 v) { u32 t = v ^ 0x80808080u; return ((t - 0x01010101u) & ~t & 0x80808080u) != 0; }
__device__ __forceinline__ u32 q4(float a, float b, float c, float d, float inv) {
  const float M = 12582912.0f;
  u32 qa = __float_as_uint(__builtin_fmaf(a, inv, M)), qb = __float_as_uint(__builtin_fmaf(b, inv, M));
  u32 qc = __float_as_uint(__builtin_fmaf(c, inv, M)), qd = __float_as_uint(__builtin_fmaf(d, inv, M));
  u32 lo = __builtin_amdgcn_perm(qb, qa, 0x0c0c0400u), hi = __builtin_amdgcn_perm(qd, qc, 0x04000c0cu);
  return lo | hi;
}
__device__ __forceinline__ u32 pack_bf16(float a, float b) {
  bf16x2 v = __builtin_convertvector(f32x2{a, b}, bf16x2);
  return __builtin_bit_cast(u32, v);
}
#define STAMP(i) do {} while (0)
constexpr int L_BASE = ((S_TOT - 4) % NS) * STAGE;
static_assert(((S_TOT - 3) % NS) == ((S_TOT - 4) % NS) + 1, "two adjacent dead slots");
static_assert(NS == 4 && (TD == 0 || (TD >= 2 && TD % 2 == 0)), "tail code");
constexpr int L_SCALE = L_BASE, L_STATE = L_BASE + 4  , L_GM = L_BASE + 20, L_PCNT = L_BASE + 24, L_PROV = L_BASE + 28, L_WAM = L_BASE + 32, L_TILE = L_BASE + 1024, TROW = 528;
}

__global__ __launch_bounds__(512, 2) void fused_gemm_a2a_q8(Params P) {
#if __HIP_DEVICE_COMPILE__
  extern __shared__ __attribute__((aligned(16))) char smem_[];
  char LDS* const smem = (char LDS*)smem_;
  const int tid = threadIdx.x, lane = tid & 63;
  const int wid = __builtin_amdgcn_readfirstlane(tid >> 6);
  const int b = blockIdx.x, grp = b & 7, idx = b >> 3;
  const int kh = grp & 1, tn = (grp >> 1) * 8 + (idx & 7), tm = idx >> 3;
  const int partner = b ^ 1;
  const int n0 = tn * BN, m0 = tm * BM, k0 = kh * (K / KSPLIT);
  const int dest = tn >> 2;
  const u16* xp = P.x; const u16* wp = P.w;
  char* myws = P.myws; char* myout = P.myout; int rank = P.rank;
  char* out_dest = P.out[dest]; char* ws_dest = P.ws[dest];
  asm volatile("" : "+s"(xp), "+s"(wp), "+s"(myws), "+s"(myout), "+s"(rank), "+s"(out_dest), "+s"(ws_dest));
  STAMP(0);
  const int rrow = wid * 8 + (lane >> 3);
  const int in_off = (grp * MS + ((idx >> 2) & 3) * BM + (rrow >> 5) * 64 + (idx >> 4) * 32 + (rrow & 31)) * NSH + (idx & 3) * BN + (lane & 7) * 16;
  auto r_myout = __builtin_amdgcn_make_buffer_rsrc((void*)myout, 0, N * NSH, 0x00020000);
  const bool recv = grp != rank;
  u32 call;
  char* ws_lane_all = nullptr;
  if (wid >= 4) {
    const int lw = wid - 4;
    u32 callv = ld_agent32(myws + WS_CALL);
    char* ws_lane = P.ws[lane & 7];
    auto rw = __builtin_amdgcn_make_buffer_rsrc((void*)wp, 0, K * N * 2, 0x00020000);
    auto rx = __builtin_amdgcn_make_buffer_rsrc((void*)xp, 0, MS * K * 2, 0x00020000);
    int vw[4], vx[4];
#pragma unroll
    for (int j = 0; j < 4; ++j) {
      int k = 16 * lw + 4 * j + (lane >> 4);
      int F = (k & 3) | ((k >> 1) & 4);
      vw[j] = k * (N * 2) + (((lane & 15) ^ (F << 1)) * 16) + n0 * 2;
      int r = (lw * 4 + j) * 8 + (lane >> 3);
      vx[j] = (m0 + r) * (K * 2) + (((lane & 7) ^ ((r >> 1) & 7)) * 16);
    }
    const int rj = (tm + tn) & 3;
    auto order = [&](int s) {
      if (s < 2) return rj + 4 * s;
      const int t = (s - 2 < rj) ? s - 2 : (s - 1 < T - 4 + rj ? s - 1 : s);
      return t + ((t & 3) == rj ? 4 : 0);
    };
    auto issue = [&](int s, int slot_off) {
      const int ss = (order(s) + grp * 4) & (T - 1);
      const int sw = (k0 + ss * BK) * (N * 2), sx = (k0 + ss * BK) * 2;
#pragma unroll
      for (int j = 0; j < 4; ++j) __builtin_amdgcn_raw_ptr_buffer_load_lds(rw, (LDS void*)(smem + slot_off + (lw * 4 + j) * 1024), 16, vw[j], sw, 0, 0);
#pragma unroll
      for (int j = 0; j < 4; ++j) __builtin_amdgcn_raw_ptr_buffer_load_lds(rx, (LDS void*)(smem + slot_off + WBYTES + (lw * 4 + j) * 1024), 16, vx[j], sx, 0, 0);
    };
    int vxk[2], lxk[2];
#pragma unroll
    for (int j = 0; j < 2; ++j) {
      const int ik = (lw >> 1) * 8 + kh * 4 + (lw & 1) * 2 + j;
      const int r = ik * 8 + (lane >> 3);
      vxk[j] = (m0 + r) * (K * 2) + (((lane & 7) ^ ((r >> 1) & 7)) * 16);
      lxk[j] = WBYTES + ik * 1024;
    }
    auto issue_half = [&](int h, int slot_off) {
      const bool own = h < TD;
      const int s = TF + (own ? h : h - TD);
      const int ss = (order(s) + (own ? grp : (partner & 7)) * 4) & (T - 1);
      const int kk = (own ? k0 : K / KSPLIT - k0) + ss * BK;
      const int sw = kk * (N * 2), sx = kk * 2;
#pragma unroll
      for (int j = 0; j < 4; ++j) __builtin_amdgcn_raw_ptr_buffer_load_lds(rw, (LDS void*)(smem + slot_off + (lw * 4 + j) * 1024), 16, vw[j], sw, 0, 0);
#pragma unroll
      for (int j = 0; j < 2; ++j) __builtin_amdgcn_raw_ptr_buffer_load_lds(rx, (LDS void*)(smem + slot_off + lxk[j]), 16, vxk[j], sx, 0, 0);
    };
#pragma unroll
    for (int s = 0; s < NS - 1; ++s) issue(s, s * STAGE);
    int slot = (NS - 1) * STAGE;
    auto next = [&]() { slot += STAGE; if (slot == NS * STAGE) slot = 0; };
#pragma unroll 1
    for (int s = 0; s + 3 < TF; ++s) { wait_vm<16>(); bar(); issue(s + 3, slot); next(); }
    if (TD == 0) {
      wait_vm<16>(); bar(); wait_vm<8>(); bar(); wait_vm<0>(); bar();
    } else {
      wait_vm<16>(); bar(); issue_half(0, slot); next();
      wait_vm<14>(); bar(); issue_half(1, slot); next();
#pragma unroll 1
      for (int h = 2; h < 2 * TD; ++h) { wait_vm<12>(); bar(); issue_half(h, slot); next(); }
      wait_vm<12>(); bar(); wait_vm<6>(); bar(); wait_vm<0>(); bar();
    }
    call = __builtin_amdgcn_readfirstlane(callv) + 1;
    ws_lane_all = ws_lane;
    if (wid == 4 && lane == 0) lds_st(smem + L_STATE, 0);
    if (wid == 4) {
      const u64 v = ld_sys64(myws + WS_GG + (size_t)(((call & 1) * REP + (idx & (RUSE - 1))) * 64 + lane) * 8);
      const bool pr = (u32)(v >> 32) == call;
      const float pv = wave_max(pr ? __uint_as_float((u32)v) : 0.f);
      const u32 pc = __builtin_popcountll(__ballot(pr));
      if (lane == 0) { lds_st(smem + L_PROV, __float_as_uint(pv)); lds_st(smem + L_PCNT, pc); }
    }
  } else {
    u32 callv = ld_agent32(myws + WS_CALL);
    {
      const u32 c0 = __builtin_amdgcn_readfirstlane(callv) + 1;
      auto rz = __builtin_amdgcn_make_buffer_rsrc((void*)(myws + WS_SPC + ((c0 + 1) & 1) * SPC_BYTES), 0, (int)SPC_BYTES, 0x00020000);
      if (recv) {
        st128<AUX_SYS>(u32x4{0, 0, 0, 0}, rz, ((grp * 32 + idx) * 8 + wid) * 1024 + lane * 16);
        st128<AUX_SYS>(u32x4{0, 0, 0, 0}, rz, ((grp * 32 + idx) * 8 + wid + 4) * 1024 + lane * 16);
      }
    }
    if (recv) {
      st128<AUX_SYS>(u32x4{0x80808080u, 0x80808080u, 0x80808080u, 0x80808080u}, r_myout, in_off);
      st128<AUX_SYS>(u32x4{0x80808080u, 0x80808080u, 0x80808080u, 0x80808080u}, r_myout, in_off + 64 * NSH);
    }
    f32x4 acc[4][4];
    const int wm = wid & 1, wn = wid >> 1;
    const int g = lane >> 4, q = (lane & 15) >> 2, p = lane & 3;
    const int F = q | ((g & 1) << 2);
    int aw[4], ax[2][2];
#pragma unroll
    for (int c = 0; c < 4; ++c) aw[c] = (8 * g + q) * 256 + ((((4 * wn + c) ^ F) << 1) | (p >> 1)) * 16 + 8 * (p & 1);
#pragma unroll
    for (int ks = 0; ks < 2; ++ks) {
      int r = wm * 64 + (lane & 15), a = WBYTES + r * 128 + (((4 * ks + g) ^ ((r >> 1) & 7)) * 16);
      ax[ks][0] = a + kh * 4096; ax[ks][1] = a + (1 - kh) * 4096;
    }
#pragma unroll
    for (int i = 0; i < 4; ++i)
#pragma unroll
      for (int j = 0; j < 4; ++j) acc[i][j] = f32x4{0, 0, 0, 0};
    s16x8 W0[4][2], X0[4][2], W1[4][2], X1[4][2];
    auto rd = [&](s16x8(&W)[4][2], s16x8(&X)[4][2], int so) {
#pragma unroll
      for (int ks = 0; ks < 2; ++ks) {
#pragma unroll
        for (int j = 0; j < 4; ++j) X[j][ks] = *(LDS s16x8*)(smem + so + ax[ks][j >> 1] + (j & 1) * 2048);
#pragma unroll
        for (int c = 0; c < 4; ++c) {
          s16x4 lo = __builtin_amdgcn_ds_read_tr16_b64_v4i16((LDS s16x4*)(smem + so + aw[c] + (32 * ks) * 256));
          s16x4 hi = __builtin_amdgcn_ds_read_tr16_b64_v4i16((LDS s16x4*)(smem + so + aw[c] + (32 * ks + 4) * 256));
          W[c][ks] = __builtin_shufflevector(lo, hi, 0, 1, 2, 3, 4, 5, 6, 7);
        }
      }
    };
    auto mm = [&](s16x8(&W)[4][2], s16x8(&X)[4][2]) {
#pragma unroll
      for (int ks = 0; ks < 2; ++ks)
#pragma unroll
        for (int c = 0; c < 4; ++c)
#pragma unroll
          for (int jj = 0; jj < 4; ++jj) {
            const int j = (c & 1) ? 3 - jj : jj;
            acc[c][j] = __builtin_amdgcn_mfma_f32_16x16x32_bf16(__builtin_bit_cast(bf16x8, W[c][ks]), __builtin_bit_cast(bf16x8, X[j][ks]), acc[c][j], 0, 0, 0);
          }
    };
    auto sched = [&]() {
#pragma unroll
      for (int i = 0; i < 32; ++i) {
        __builtin_amdgcn_sched_group_barrier(0x8, 1, 0);
        if (i < 24) __builtin_amdgcn_sched_group_barrier(0x100, 1, 0);
      }
    };
    auto rdh = [&](s16x8(&W)[4][2], s16x8(&X)[4][2], int so) {
#pragma unroll
      for (int ks = 0; ks < 2; ++ks) {
#pragma unroll
        for (int j = 0; j < 2; ++j) X[j][ks] = *(LDS s16x8*)(smem + so + ax[ks][0] + j * 2048);
#pragma unroll
        for (int c = 0; c < 4; ++c) {
          s16x4 lo = __builtin_amdgcn_ds_read_tr16_b64_v4i16((LDS s16x4*)(smem + so + aw[c] + (32 * ks) * 256));
          s16x4 hi = __builtin_amdgcn_ds_read_tr16_b64_v4i16((LDS s16x4*)(smem + so + aw[c] + (32 * ks + 4) * 256));
          W[c][ks] = __builtin_shufflevector(lo, hi, 0, 1, 2, 3, 4, 5, 6, 7);
        }
      }
    };
    auto mmj = [&](s16x8(&W)[4][2], s16x8(&X)[4][2], int j0) {
#pragma unroll
      for (int ks = 0; ks < 2; ++ks)
#pragma unroll
        for (int c = 0; c < 4; ++c)
#pragma unroll
          for (int j = j0; j < j0 + 2; ++j)
            acc[c][j] = __builtin_amdgcn_mfma_f32_16x16x32_bf16(__builtin_bit_cast(bf16x8, W[c][ks]), __builtin_bit_cast(bf16x8, X[j][ks]), acc[c][j], 0, 0, 0);
    };
    auto schedh = [&]() {
#pragma unroll
      for (int i = 0; i < 16; ++i) {
        __builtin_amdgcn_sched_group_barrier(0x8, 1, 0);
        if (i < 10) __builtin_amdgcn_sched_group_barrier(0x100, 2, 0);
      }
    };
    auto rs = __builtin_amdgcn_make_buffer_rsrc((void*)(myws + WS_SLAB), 0, NWG * 32768, 0x00020000);
    const int so_out = (partner * 4 + wid) * 8192, so_in = (b * 4 + wid) * 8192;
    u32 tag;
    constexpr int NX = 4;
    u32 in_sc;
    auto send_partial = [&]() {
      call = __builtin_amdgcn_readfirstlane(callv) + 1;
      tag = call & 1;
      float pm = 0.f;
#pragma unroll
      for (int c = 0; c < 4; ++c)
#pragma unroll
        for (int j = 2; j < 4; ++j)
#pragma unroll
          for (int r = 0; r < 4; ++r) pm = fmaxf(pm, fabsf(acc[c][j][r]));
      u32 eb = __float_as_uint(pm) & 0x7f800000u;
      eb = min(max(eb, 15u << 23), 254u << 23);
      const float up = __uint_as_float((268u << 23) - eb);
      const u32 dn = eb - (14u << 23);
#pragma unroll
      for (int c = 0; c < 4; ++c) {
        u32x4 e;
#pragma unroll
        for (int j = 0; j < 2; ++j)
#pragma unroll
          for (int r = 0; r < 2; ++r) {
            f16x2 h = __builtin_convertvector(f32x2{acc[c][2 + j][2 * r] * up, acc[c][2 + j][2 * r + 1] * up}, f16x2);
            e[j * 2 + r] = (__builtin_bit_cast(u32, h) & ~1u) | tag;
          }
        st128<AUX_SC1>(e, rs, lane * 16 + c * 1024 + so_out);
      }
      __builtin_amdgcn_raw_buffer_store_b32(dn | tag, rs, lane * 4 + 4096, so_out, AUX_SC1);
      STAMP(2);
    };
    u32x4 in[NX];
    auto load_partial = [&]() {
#pragma unroll
      for (int i = 0; i < NX; ++i) in[i] = __builtin_amdgcn_raw_buffer_load_b128(rs, lane * 16 + i * 1024, so_in, AUX_SC1);
      in_sc = __builtin_amdgcn_raw_buffer_load_b32(rs, lane * 4 + 4096, so_in, AUX_SC1);
    };
    int slot = 0;
    auto next = [&]() { slot += STAGE; if (slot == NS * STAGE) slot = 0; };
    bar();
    rd(W0, X0, slot); next();
    for (int s = 1; s < TF - 1; s += 2) {
      bar_lgkm();
      rd(W1, X1, slot); mm(W0, X0); sched(); next();
      bar_lgkm();
      rd(W0, X0, slot); mm(W1, X1); sched(); next();
    }
    bar_lgkm();
    rd(W1, X1, slot); mm(W0, X0); sched(); next();
    if (TD == 0) {
      mmj(W1, X1, 2); send_partial(); mmj(W1, X1, 0);
      load_partial();
    } else {
      bar_lgkm();
      rdh(W0, X0, slot); mmj(W1, X1, 2); schedh(); next();
      send_partial();
      mmj(W1, X1, 0);
#pragma unroll 1
      for (int h = 1; h < 2 * TD - 3; h += 2) {
        bar_lgkm();
        rdh(W1, X1, slot); mmj(W0, X0, 0); schedh(); next();
        bar_lgkm();
        rdh(W0, X0, slot); mmj(W1, X1, 0); schedh(); next();
      }
      bar_lgkm();
      rdh(W1, X1, slot); mmj(W0, X0, 0); schedh(); next();
      bar_lgkm();
      load_partial();
      rdh(W0, X0, slot); mmj(W1, X1, 0); schedh(); next();
      bar_lgkm();
      rdh(W1, X1, slot); mmj(W0, X0, 0); schedh();
      mmj(W1, X1, 0);
    }
    STAMP(1);
    bool ok = true;
    {
      u64 t0 = __builtin_amdgcn_s_memrealtime();
      u32 tm_ = 0;
      for (;;) {
        u32 an = in_sc, orr = in_sc;
#pragma unroll
        for (int i = 0; i < NX; ++i) { an &= (in[i][0] & in[i][1]) & (in[i][2] & in[i][3]); orr |= (in[i][0] | in[i][1]) | (in[i][2] | in[i][3]); }
        if (__all(((tag ? an : ~orr) & 1u) != 0)) break;
        if (give_up(myws, call, tm_, t0)) { ok = false; break; }
        asm volatile("" ::: "memory");
        load_partial();
        tm_ = ld_agent32(myws + WS_TMO);
      }
    }
    STAMP(3);
    float am = 0.f;
#pragma unroll
    for (int c = 0; c < 4; ++c)
#pragma unroll
      for (int j = 0; j < 2; ++j) {
        const float dn = __uint_as_float(in_sc & ~1u);
#pragma unroll
        for (int r = 0; r < 2; ++r) {
          float a0 = acc[c][j][2 * r], a1 = acc[c][j][2 * r + 1];
          asm("v_fma_mix_f32 %0, %1, %2, %0 op_sel_hi:[1,0,0]" : "+v"(a0) : "v"(in[c][j * 2 + r]), "v"(dn));
          asm("v_fma_mix_f32 %0, %1, %2, %0 op_sel:[1,0,0] op_sel_hi:[1,0,0]" : "+v"(a1) : "v"(in[c][j * 2 + r]), "v"(dn));
          acc[c][j][2 * r] = a0; acc[c][j][2 * r + 1] = a1;
        }
#pragma unroll
        for (int r = 0; r < 4; ++r) am = fmaxf(am, fabsf(acc[c][j][r]));
      }
    am = wave_max(am);
    if (lane == 0 && ok) {
      const u64 gr = ((u64)call << 32) | __float_as_uint(am);
      st_agent64(myws + WS_L1G + (size_t)((call & 1) * 8 + grp) * 1024 + (idx * 4 + wid) * 8, gr);
    }
    if (lane == 0) lds_st(smem + L_WAM + wid * 4, __float_as_uint(am));
#pragma unroll
    for (int c = 0; c < 4; ++c)
#pragma unroll
      for (int j = 0; j < 2; ++j) {
        int row = wm * 32 + j * 16 + (lane & 15), col = wn * 64 + c * 16 + 4 * g;
        *(LDS f32x4*)(smem + L_TILE + row * TROW + col * 4) = acc[c][j];
      }
    STAMP(4);
  }
  bar_lgkm();
  const int par = call & 1;
  f32x4 yv[4];
#pragma unroll
  for (int i = 0; i < 4; ++i) yv[i] = *(LDS f32x4*)(smem + L_TILE + rrow * TROW + (lane & 7) * 64 + i * 16);
  auto r_stg_in = __builtin_amdgcn_make_buffer_rsrc((void*)(myws + WS_STG + par * STG_BYTES), 0, (int)STG_BYTES, 0x00020000);
  const int stg_in_off = ((grp * 32 + idx) * 8 + wid) * 2048 + lane * 16;
  const int stg_probe = ((grp * 32 + idx) * 8 + wid) * 2048 + 2044;
  bool sent = false, got = false, rearmed = false;
  u32 sent_a = 0, got_a = 0;
  auto r_spc_in = __builtin_amdgcn_make_buffer_rsrc((void*)(myws + WS_SPC + par * SPC_BYTES), 0, (int)SPC_BYTES, 0x00020000);
  const int spc_in_off = ((grp * 32 + idx) * 8 + wid) * 1024 + lane * 16;
  const char* hdr_in = myws + WS_HDR + (size_t)(par * 2048 + (grp * 32 + idx) * 8 + wid) * 8;
  auto nzb4 = [](u32x4 v) { auto z = [](u32 t) { return ((t - 0x01010101u) & ~t & 0x80808080u) != 0; }; return !(z(v[0]) || z(v[1]) || z(v[2]) || z(v[3])); };
  u32x4 e0, e1;
  u64 t0 = __builtin_amdgcn_s_memrealtime();
  const bool can_early = EARLY && dest != rank;
  auto send_early = [&]() {
    auto r_stg = __builtin_amdgcn_make_buffer_rsrc((void*)(ws_dest + WS_STG + par * STG_BYTES), 0, (int)STG_BYTES, 0x00020000);
    const int so = ((rank * 32 + ((tn & 3) + 4 * tm + 16 * kh)) * 8 + wid) * 2048;
    u32x4 a = {enc(pack_bf16(yv[0][0], yv[0][1])), enc(pack_bf16(yv[0][2], yv[0][3])), enc(pack_bf16(yv[1][0], yv[1][1])), enc(pack_bf16(yv[1][2], yv[1][3]))};
    u32x4 c = {enc(pack_bf16(yv[2][0], yv[2][1])), enc(pack_bf16(yv[2][2], yv[2][3])), enc(pack_bf16(yv[3][0], yv[3][1])), enc(pack_bf16(yv[3][2], yv[3][3]))};
    st128<AUX_SYS>(a, r_stg, lane * 16 + so);
    st128<AUX_SYS>(c, r_stg, lane * 16 + 1024 + so);
    sent = true;
    STAMP(9);
  };
  auto send_spec = [&]() {
    u32 am = max(max(lds_ld(smem + L_PROV), max(lds_ld(smem + L_WAM), lds_ld(smem + L_WAM + 4))), max(lds_ld(smem + L_WAM + 8), lds_ld(smem + L_WAM + 12)));
    am = __builtin_amdgcn_readfirstlane(am);
    const float pa = __uint_as_float(am), pinv = pa > 0.f ? 127.0f / pa : 0.f;
    u32x4 o = {q4(yv[0][0], yv[0][1], yv[0][2], yv[0][3], pinv), q4(yv[1][0], yv[1][1], yv[1][2], yv[1][3], pinv),
               q4(yv[2][0], yv[2][1], yv[2][2], yv[2][3], pinv), q4(yv[3][0], yv[3][1], yv[3][2], yv[3][3], pinv)};
    const int ch = (rank * 32 + ((tn & 3) + 4 * tm + 16 * kh)) * 8 + wid;
    auto r_spc = __builtin_amdgcn_make_buffer_rsrc((void*)(ws_dest + WS_SPC + par * SPC_BYTES), 0, (int)SPC_BYTES, 0x00020000);
    st128<AUX_SYS>(o ^ 0x80808080u, r_spc, ch * 1024 + lane * 16);
    if (lane == 0) st_sys64(ws_dest + WS_HDR + (size_t)(par * 2048 + ch) * 8, ((u64)call << 32) | am);
    sent = true; sent_a = am + 1;
    STAMP(10);
  };
  if (wid == 4) {
    const char* gsrc = myws + WS_GG + (size_t)((par * REP + (idx & (RUSE - 1))) * 64 + lane) * 8;
    u64 v; bool ok = true;
    for (;;) {
      v = ld_sys64(gsrc);
      u32 tm_ = ld_agent32(myws + WS_TMO);
      if (__all((u32)(v >> 32) == call)) { ok = tm_ != call; break; }
      if (give_up(myws, call, tm_, t0)) { ok = false; break; }
      {
        const bool pr = (u32)(v >> 32) == call;
        const float pv = wave_max(pr ? __uint_as_float((u32)v) : 0.f);
        const u32 pc = __builtin_popcountll(__ballot(pr));
        if (lane == 0) { lds_st(smem + L_PROV, __float_as_uint(pv)); __builtin_amdgcn_s_waitcnt(0xC07F); lds_st(smem + L_PCNT, pc); }
      }
    }
    float gm = wave_max(__uint_as_float((u32)v));
    if (lane == 0) {
      lds_st(smem + L_SCALE, __float_as_uint(gm > 0.f ? 127.0f / gm : 0.f));
      lds_st(smem + L_GM, __float_as_uint(gm));
      __builtin_amdgcn_s_waitcnt(0xC07F);
      lds_st(smem + L_STATE, ok ? 1u : 2u);
      if (b == 0) st_agent32(myws + WS_CALL, call);
    }
    STAMP(6);
  } else {
    if (wid == 5 && idx == 0) {
      const char* l1 = myws + WS_L1G + (size_t)(par * 8 + grp) * 1024 + lane * 16;
      char* dst = ws_lane_all + WS_GG + (size_t)((par * REP + (lane >> 3)) * 64 + rank * 8 + grp) * 8;
      auto rl = __builtin_amdgcn_make_buffer_rsrc((void*)myws, 0, 1 << 20, 0x00020000);
      for (;;) {
        asm volatile("" ::: "memory");
        u32x4 v = __builtin_amdgcn_raw_buffer_load_b128(rl, (int)(l1 - myws), 0, AUX_SC1);
        u32 tm_ = ld_agent32(myws + WS_TMO);
        if (__all(v[1] == call && v[3] == call)) {
          float gm = wave_max(fmaxf(__uint_as_float(v[0]), __uint_as_float(v[2])));
          if ((lane >> 3) < RUSE) st_sys64(dst, ((u64)call << 32) | __float_as_uint(gm));
          break;
        }
        if (give_up(myws, call, tm_, t0)) break;
      }
      STAMP(5);
    }
    const u64 due = t0 + (u64)(wid - (wid > 4)) * PACE;
    while (__builtin_amdgcn_readfirstlane(lds_ld(smem + L_STATE)) == 0) {
      if (can_early && !sent && __builtin_amdgcn_readfirstlane(lds_ld(smem + L_PCNT)) >= SPEC && __builtin_amdgcn_s_memrealtime() >= t0 + (u64)(wid - (wid > 4)) * SPACE) {
        send_spec();
      } else
      if (can_early && !sent && __builtin_amdgcn_s_memrealtime() >= due) {
        send_early();
      } else if (EARLY && recv && !got && got_a == 0 && (sent || !can_early)) {
        asm volatile("" ::: "memory");
        const u32 ps = __builtin_amdgcn_raw_buffer_load_b32(r_stg_in, stg_probe, 0, AUX_SYS);
        const u64 hd = ld_sys64(hdr_in);
        if (__builtin_amdgcn_readfirstlane((u32)(hd >> 32)) == call) {
          e0 = __builtin_amdgcn_raw_buffer_load_b128(r_spc_in, spc_in_off, 0, AUX_SYS);
          if (__all(nzb4(e0))) got_a = __builtin_amdgcn_readfirstlane((u32)hd) + 1;
        } else
        if (__builtin_amdgcn_readfirstlane(ps) != 0) {
          e0 = __builtin_amdgcn_raw_buffer_load_b128(r_stg_in, stg_in_off, 0, AUX_SYS);
          e1 = __builtin_amdgcn_raw_buffer_load_b128(r_stg_in, stg_in_off + 1024, 0, AUX_SYS);
          got = __all(nz4(e0) && nz4(e1));
          if (got) {
            rearmed = true;
            st128<AUX_SYS>(u32x4{0, 0, 0, 0}, r_stg_in, stg_in_off);
            st128<AUX_SYS>(u32x4{0, 0, 0, 0}, r_stg_in, stg_in_off + 1024);
          }
        } else __builtin_amdgcn_s_sleep(PRE_SLEEP);
      } else {
        __builtin_amdgcn_s_sleep(1);
      }
    }
  }
  __builtin_amdgcn_s_waitcnt(0xC07F);
  const u32 state = __builtin_amdgcn_readfirstlane(lds_ld(smem + L_STATE));
  const float inv = __uint_as_float(lds_ld(smem + L_SCALE));
  const u32 gm1 = __builtin_amdgcn_readfirstlane(lds_ld(smem + L_GM)) + 1;
  if (state == 1) {
    if (sent_a != 0 && sent_a != gm1) { wait_vm<0>(); sent = false; }
    if (!sent) {
      u32x4 o = {q4(yv[0][0], yv[0][1], yv[0][2], yv[0][3], inv), q4(yv[1][0], yv[1][1], yv[1][2], yv[1][3], inv),
                 q4(yv[2][0], yv[2][1], yv[2][2], yv[2][3], inv), q4(yv[3][0], yv[3][1], yv[3][2], yv[3][3], inv)};
      const int grow = rank * MS + m0 + (rrow >> 5) * 64 + kh * 32 + (rrow & 31);
      auto ro = __builtin_amdgcn_make_buffer_rsrc((void*)out_dest, 0, N * NSH, 0x00020000);
      st128<AUX_SYS>(o, ro, grow * NSH + (tn & 3) * BN + (lane & 7) * 16);
    }
    STAMP(7);
    if (recv) {
      while (!got && got_a != gm1) {
        asm volatile("" ::: "memory");
        u32x4 o = __builtin_amdgcn_raw_buffer_load_b128(r_myout, in_off, 0, AUX_SYS);
        const u64 hd = ld_sys64(hdr_in);
        u32 ps = __builtin_amdgcn_raw_buffer_load_b32(r_stg_in, stg_probe, 0, AUX_SYS);
        u32 tm_ = ld_agent32(myws + WS_TMO);
        if (__all(!(has80(o[0]) || has80(o[1]) || has80(o[2]) || has80(o[3])))) break;
        if (__builtin_amdgcn_readfirstlane((u32)(hd >> 32)) == call && __builtin_amdgcn_readfirstlane((u32)hd) + 1 == gm1) {
          e0 = __builtin_amdgcn_raw_buffer_load_b128(r_spc_in, spc_in_off, 0, AUX_SYS);
          if (__all(nzb4(e0))) { got_a = gm1; break; }
        } else
        if (__builtin_amdgcn_readfirstlane(ps) != 0) {
          e0 = __builtin_amdgcn_raw_buffer_load_b128(r_stg_in, stg_in_off, 0, AUX_SYS);
          e1 = __builtin_amdgcn_raw_buffer_load_b128(r_stg_in, stg_in_off + 1024, 0, AUX_SYS);
          got = __all(nz4(e0) && nz4(e1));
        }
        if (!got && give_up(myws, call, tm_, t0)) break;
      }
      if (got_a == gm1) st128<AUX_SYS>(e0 ^ 0x80808080u, r_myout, in_off);
      if (EARLY && got) {
        e0 -= 1u; e1 -= 1u;
        auto lo = [](u32 v) { return __uint_as_float(v << 16); };
        auto hi = [](u32 v) { return __uint_as_float(v & 0xffff0000u); };
        u32x4 o = {q4(lo(e0[0]), hi(e0[0]), lo(e0[1]), hi(e0[1]), inv), q4(lo(e0[2]), hi(e0[2]), lo(e0[3]), hi(e0[3]), inv),
                   q4(lo(e1[0]), hi(e1[0]), lo(e1[1]), hi(e1[1]), inv), q4(lo(e1[2]), hi(e1[2]), lo(e1[3]), hi(e1[3]), inv)};
        st128<AUX_SYS>(o, r_myout, in_off);
        if (!rearmed) {
          st128<AUX_SYS>(u32x4{0, 0, 0, 0}, r_stg_in, stg_in_off);
          st128<AUX_SYS>(u32x4{0, 0, 0, 0}, r_stg_in, stg_in_off + 1024);
        }
      }
      STAMP(8);
    }
  } else if (dest == rank) {
    const int grow = rank * MS + m0 + (rrow >> 5) * 64 + kh * 32 + (rrow & 31);
    st128<AUX_SYS>(u32x4{0x80808080u, 0x80808080u, 0x80808080u, 0x80808080u}, r_myout, grow * NSH + (tn & 3) * BN + (lane & 7) * 16);
  }
#endif
}

extern "C" void kernel_launch(int rank, int nranks, ncclComm_t comm, hipStream_t stream, void* const* d_in, const int* in_sizes, int n_in,
                              void* d_out, int out_size, const int* in_dtypes, int out_dtype, void* const* d_out_all,
                              void* const* d_ws_all, size_t ws_bytes) {
  if (nranks != NR || (unsigned)rank >= (unsigned)NR || ws_bytes < WS_NEED || n_in < 2 || in_sizes[0] != MS * K || in_sizes[1] != K * N || out_size != N * NSH) {
    fprintf(stderr, "unexpected problem shape\n");
    return;
  }
  Params P;
  P.x = (const u16*)d_in[0];
  P.w = (const u16*)d_in[1];
  for (int p = 0; p < NR; ++p) { P.out[p] = (char*)d_out_all[p]; P.ws[p] = (char*)d_ws_all[p]; }
  P.myws = P.ws[rank]; P.myout = (char*)d_out;
  P.rank = rank;
  hipError_t e = hipFuncSetAttribute((const void*)fused_gemm_a2a_q8, hipFuncAttributeMaxDynamicSharedMemorySize, LDS_BYTES);
  if (e != hipSuccess) { fprintf(stderr, "hipFuncSetAttribute: %s\n", hipGetErrorString(e)); return; }
  hipLaunchKernelGGL(fused_gemm_a2a_q8, dim3(NWG), dim3(512), LDS_BYTES, stream, P);
}
```

```cpp
#include <hip/hip_runtime.h>
#include <rccl/rccl.h>
#include <cstdio>

typedef unsigned short u16;
typedef unsigned int u32;
typedef unsigned long long u64;
typedef __attribute__((ext_vector_type(8))) short s16x8;
typedef __attribute__((ext_vector_type(4))) short s16x4;
typedef __attribute__((ext_vector_type(8))) __bf16 bf16x8;
typedef __attribute__((ext_vector_type(2))) __bf16 bf16x2;
typedef __attribute__((ext_vector_type(4))) float f32x4;
typedef __attribute__((ext_vector_type(2))) float f32x2;
typedef __attribute__((ext_vector_type(2))) _Float16 f16x2;
typedef __attribute__((ext_vector_type(4))) unsigned int u32x4;
#define LDS __attribute__((address_space(3)))
#define GAS __attribute__((address_space(1)))

namespace {
constexpr int EARLY = 1;
constexpr int PACE = 160;
constexpr int TAILD = 2;
constexpr int PRE_SLEEP = 4;
constexpr int NR = 8;
constexpr int MS = 512, N = 4096, K = 4096;
constexpr int NSH = N / NR;
constexpr int BM = 128, BN = 128, BK = 64, KSPLIT = 2, NS = 4;
constexpr int WBYTES = BK * BN * 2, XBYTES = BM * BK * 2, STAGE = WBYTES + XBYTES;
constexpr int T = K / KSPLIT / BK;
constexpr int TD = TAILD;
constexpr int TF = T - TD, S_TOT = T + TD;
constexpr int NWG = 256;
constexpr int LDS_BYTES = NS * STAGE;
constexpr int LPW = 8;
constexpr size_t WS_CALL = 0;
constexpr size_t WS_TMO = 128;
constexpr size_t WS_L1G = 4096;
constexpr int REP = 8, RUSE = 4;
constexpr size_t WS_GG = WS_L1G + 2 * 8 * 128 * 8;
constexpr size_t WS_DBG = 512 * 1024;
constexpr size_t WS_SLAB = 1 << 20;
constexpr size_t WS_STG = WS_SLAB + (size_t)NWG * 32768;
constexpr size_t STG_BYTES = (size_t)N * NSH * 2;
constexpr size_t WS_NEED = WS_STG + 2 * STG_BYTES;
constexpr u64 DEADLINE = 100000000ull;
constexpr int AUX_SC1 = 16, AUX_SYS = 17;

struct Params {
  const u16* x; const u16* w;
  char* myws; char* myout;
  char* out[NR]; char* ws[NR];
  int rank;
};

template <int N_> __device__ __forceinline__ void wait_vm() { asm volatile("s_waitcnt vmcnt(%0)" ::"i"(N_) : "memory"); }
__device__ __forceinline__ void bar() { asm volatile("s_barrier" ::: "memory"); }
__device__ __forceinline__ void bar_lgkm() {
  __builtin_amdgcn_sched_barrier(0);
  __builtin_amdgcn_s_waitcnt(0xC07F);
  asm volatile("s_barrier" ::: "memory");
  __builtin_amdgcn_sched_barrier(0);
}
__device__ __forceinline__ u32 ld_agent32(const char* p) { return __hip_atomic_load((const GAS u32*)p, __ATOMIC_RELAXED, __HIP_MEMORY_SCOPE_AGENT); }
__device__ __forceinline__ u64 ld_sys64(const char* p) { return __hip_atomic_load((const GAS u64*)p, __ATOMIC_RELAXED, __HIP_MEMORY_SCOPE_SYSTEM); }
__device__ __forceinline__ void st_agent32(char* p, u32 v) { __hip_atomic_store((GAS u32*)p, v, __ATOMIC_RELAXED, __HIP_MEMORY_SCOPE_AGENT); }
__device__ __forceinline__ void st_agent64(char* p, u64 v) { __hip_atomic_store((GAS u64*)p, v, __ATOMIC_RELAXED, __HIP_MEMORY_SCOPE_AGENT); }
__device__ __forceinline__ void st_sys64(char* p, u64 v) { __hip_atomic_store((GAS u64*)p, v, __ATOMIC_RELAXED, __HIP_MEMORY_SCOPE_SYSTEM); }
__device__ __forceinline__ u32 lds_ld(const char LDS* p) { return __hip_atomic_load((const LDS u32*)p, __ATOMIC_RELAXED, __HIP_MEMORY_SCOPE_WORKGROUP); }
__device__ __forceinline__ void lds_st(char LDS* p, u32 v) { __hip_atomic_store((LDS u32*)p, v, __ATOMIC_RELAXED, __HIP_MEMORY_SCOPE_WORKGROUP); }

__device__ __forceinline__ bool give_up(char* myws, u32 call, u32 tmo, u64 t0) {
  bool bad = tmo == call || (__builtin_amdgcn_s_memrealtime() - t0) > DEADLINE;
  bad = __builtin_amdgcn_readfirstlane(bad);
  if (bad) st_agent32(myws + WS_TMO, call);
  return bad;
}
__device__ __forceinline__ float wave_max(float v) {
  int x = __float_as_int(v);
  x = max(x, __builtin_amdgcn_update_dpp(0, x, 0x111, 0xf, 0xf, false));
  x = max(x, __builtin_amdgcn_update_dpp(0, x, 0x112, 0xf, 0xf, false));
  x = max(x, __builtin_amdgcn_update_dpp(0, x, 0x114, 0xf, 0xf, false));
  x = max(x, __builtin_amdgcn_update_dpp(0, x, 0x118, 0xf, 0xf, false));
  int a = __builtin_amdgcn_readlane(x, 15), b = __builtin_amdgcn_readlane(x, 31), c = __builtin_amdgcn_readlane(x, 47), d = __builtin_amdgcn_readlane(x, 63);
  return __int_as_float(max(max(a, b), max(c, d)));
}
template <int AUX, typename R> __device__ __forceinline__ void st128(u32x4 v, R rsrc, int voff) { __builtin_amdgcn_raw_buffer_store_b128(v, rsrc, voff, 0, AUX); }
__device__ __forceinline__ u32 enc(u32 v) { return __builtin_elementwise_add_sat(v, 1u); }
__device__ __forceinline__ bool nz4(u32x4 v) { return v[0] != 0 && v[1] != 0 && v[2] != 0 && v[3] != 0; }
__device__ __forceinline__ bool has80(u32 v) { u32 t = v ^ 0x80808080u; return ((t - 0x01010101u) & ~t & 0x80808080u) != 0; }
__device__ __forceinline__ u32 q4(float a, float b, float c, float d, float inv) {
  const float M = 12582912.0f;
  u32 qa = __float_as_uint(__builtin_fmaf(a, inv, M)), qb = __float_as_uint(__builtin_fmaf(b, inv, M));
  u32 qc = __float_as_uint(__builtin_fmaf(c, inv, M)), qd = __float_as_uint(__builtin_fmaf(d, inv, M));
  u32 lo = __builtin_amdgcn_perm(qb, qa, 0x0c0c0400u), hi = __builtin_amdgcn_perm(qd, qc, 0x04000c0cu);
  return lo | hi;
}
__device__ __forceinline__ u32 pack_bf16(float a, float b) {
  bf16x2 v = __builtin_convertvector(f32x2{a, b}, bf16x2);
  return __builtin_bit_cast(u32, v);
}
#define STAMP(i) do {} while (0)
constexpr int L_BASE = ((S_TOT - 4) % NS) * STAGE;
static_assert(((S_TOT - 3) % NS) == ((S_TOT - 4) % NS) + 1, "two adjacent dead slots");
static_assert(NS == 4 && (TD == 0 || (TD >= 2 && TD % 2 == 0)), "tail code");
constexpr int L_SCALE = L_BASE, L_STATE = L_BASE + 4  , L_TILE = L_BASE + 1024, TROW = 528;
}

__global__ __launch_bounds__(512, 2) void fused_gemm_a2a_q8(Params P) {
#if __HIP_DEVICE_COMPILE__
  extern __shared__ __attribute__((aligned(16))) char smem_[];
  char LDS* const smem = (char LDS*)smem_;
  const int tid = threadIdx.x, lane = tid & 63;
  const int wid = __builtin_amdgcn_readfirstlane(tid >> 6);
  const int b = blockIdx.x, grp = b & 7, idx = b >> 3;
  const int kh = grp & 1, tn = (grp >> 1) * 8 + (idx & 7), tm = idx >> 3;
  const int partner = b ^ 1;
  const int n0 = tn * BN, m0 = tm * BM, k0 = kh * (K / KSPLIT);
  const int dest = tn >> 2;
  const u16* xp = P.x; const u16* wp = P.w;
  char* myws = P.myws; char* myout = P.myout; int rank = P.rank;
  char* out_dest = P.out[dest]; char* ws_dest = P.ws[dest];
  asm volatile("" : "+s"(xp), "+s"(wp), "+s"(myws), "+s"(myout), "+s"(rank), "+s"(out_dest), "+s"(ws_dest));
  STAMP(0);
  const int rrow = wid * 8 + (lane >> 3);
  const int in_off = (grp * MS + ((idx >> 2) & 3) * BM + (rrow >> 5) * 64 + (idx >> 4) * 32 + (rrow & 31)) * NSH + (idx & 3) * BN + (lane & 7) * 16;
  auto r_myout = __builtin_amdgcn_make_buffer_rsrc((void*)myout, 0, N * NSH, 0x00020000);
  const bool recv = grp != rank;
  u32 call;
  char* ws_lane_all = nullptr;
  if (wid >= 4) {
    const int lw = wid - 4;
    u32 callv = ld_agent32(myws + WS_CALL);
    char* ws_lane = P.ws[lane & 7];
    auto rw = __builtin_amdgcn_make_buffer_rsrc((void*)wp, 0, K * N * 2, 0x00020000);
    auto rx = __builtin_amdgcn_make_buffer_rsrc((void*)xp, 0, MS * K * 2, 0x00020000);
    int vw[4], vx[4];
#pragma unroll
    for (int j = 0; j < 4; ++j) {
      int k = 16 * lw + 4 * j + (lane >> 4);
      int F = (k & 3) | ((k >> 1) & 4);
      vw[j] = k * (N * 2) + (((lane & 15) ^ (F << 1)) * 16) + n0 * 2;
      int r = (lw * 4 + j) * 8 + (lane >> 3);
      vx[j] = (m0 + r) * (K * 2) + (((lane & 7) ^ ((r >> 1) & 7)) * 16);
    }
    const int rj = (tm + tn) & 3;
    auto order = [&](int s) {
      if (s < 2) return rj + 4 * s;
      const int t = (s - 2 < rj) ? s - 2 : (s - 1 < T - 4 + rj ? s - 1 : s);
      return t + ((t & 3) == rj ? 4 : 0);
    };
    auto issue = [&](int s, int slot_off) {
      const int ss = (order(s) + grp * 4) & (T - 1);
      const int sw = (k0 + ss * BK) * (N * 2), sx = (k0 + ss * BK) * 2;
#pragma unroll
      for (int j = 0; j < 4; ++j) __builtin_amdgcn_raw_ptr_buffer_load_lds(rw, (LDS void*)(smem + slot_off + (lw * 4 + j) * 1024), 16, vw[j], sw, 0, 0);
#pragma unroll
      for (int j = 0; j < 4; ++j) __builtin_amdgcn_raw_ptr_buffer_load_lds(rx, (LDS void*)(smem + slot_off + WBYTES + (lw * 4 + j) * 1024), 16, vx[j], sx, 0, 0);
    };
    int vxk[2], lxk[2];
#pragma unroll
    for (int j = 0; j < 2; ++j) {
      const int ik = (lw >> 1) * 8 + kh * 4 + (lw & 1) * 2 + j;
      const int r = ik * 8 + (lane >> 3);
      vxk[j] = (m0 + r) * (K * 2) + (((lane & 7) ^ ((r >> 1) & 7)) * 16);
      lxk[j] = WBYTES + ik * 1024;
    }
    auto issue_half = [&](int h, int slot_off) {
      const bool own = h < TD;
      const int s = TF + (own ? h : h - TD);
      const int ss = (order(s) + (own ? grp : (partner & 7)) * 4) & (T - 1);
      const int kk = (own ? k0 : K / KSPLIT - k0) + ss * BK;
      const int sw = kk * (N * 2), sx = kk * 2;
#pragma unroll
      for (int j = 0; j < 4; ++j) __builtin_amdgcn_raw_ptr_buffer_load_lds(rw, (LDS void*)(smem + slot_off + (lw * 4 + j) * 1024), 16, vw[j], sw, 0, 0);
#pragma unroll
      for (int j = 0; j < 2; ++j) __builtin_amdgcn_raw_ptr_buffer_load_lds(rx, (LDS void*)(smem + slot_off + lxk[j]), 16, vxk[j], sx, 0, 0);
    };
#pragma unroll
    for (int s = 0; s < NS - 1; ++s) issue(s, s * STAGE);
    int slot = (NS - 1) * STAGE;
    auto next = [&]() { slot += STAGE; if (slot == NS * STAGE) slot = 0; };
#pragma unroll 1
    for (int s = 0; s + 3 < TF; ++s) { wait_vm<16>(); bar(); issue(s + 3, slot); next(); }
    if (TD == 0) {
      wait_vm<16>(); bar(); wait_vm<8>(); bar(); wait_vm<0>(); bar();
    } else {
      wait_vm<16>(); bar(); issue_half(0, slot); next();
      wait_vm<14>(); bar(); issue_half(1, slot); next();
#pragma unroll 1
      for (int h = 2; h < 2 * TD; ++h) { wait_vm<12>(); bar(); issue_half(h, slot); next(); }
      wait_vm<12>(); bar(); wait_vm<6>(); bar(); wait_vm<0>(); bar();
    }
    call = __builtin_amdgcn_readfirstlane(callv) + 1;
    ws_lane_all = ws_lane;
    if (wid == 4 && lane == 0) lds_st(smem + L_STATE, 0);
  } else {
    u32 callv = ld_agent32(myws + WS_CALL);
    if (recv) {
      st128<AUX_SYS>(u32x4{0x80808080u, 0x80808080u, 0x80808080u, 0x80808080u}, r_myout, in_off);
      st128<AUX_SYS>(u32x4{0x80808080u, 0x80808080u, 0x80808080u, 0x80808080u}, r_myout, in_off + 64 * NSH);
    }
    f32x4 acc[4][4];
    const int wm = wid & 1, wn = wid >> 1;
    const int g = lane >> 4, q = (lane & 15) >> 2, p = lane & 3;
    const int F = q | ((g & 1) << 2);
    int aw[4], ax[2][2];
#pragma unroll
    for (int c = 0; c < 4; ++c) aw[c] = (8 * g + q) * 256 + ((((4 * wn + c) ^ F) << 1) | (p >> 1)) * 16 + 8 * (p & 1);
#pragma unroll
    for (int ks = 0; ks < 2; ++ks) {
      int r = wm * 64 + (lane & 15), a = WBYTES + r * 128 + (((4 * ks + g) ^ ((r >> 1) & 7)) * 16);
      ax[ks][0] = a + kh * 4096; ax[ks][1] = a + (1 - kh) * 4096;
    }
#pragma unroll
    for (int i = 0; i < 4; ++i)
#pragma unroll
      for (int j = 0; j < 4; ++j) acc[i][j] = f32x4{0, 0, 0, 0};
    s16x8 W0[4][2], X0[4][2], W1[4][2], X1[4][2];
    auto rd = [&](s16x8(&W)[4][2], s16x8(&X)[4][2], int so) {
#pragma unroll
      for (int ks = 0; ks < 2; ++ks) {
#pragma unroll
        for (int j = 0; j < 4; ++j) X[j][ks] = *(LDS s16x8*)(smem + so + ax[ks][j >> 1] + (j & 1) * 2048);
#pragma unroll
        for (int c = 0; c < 4; ++c) {
          s16x4 lo = __builtin_amdgcn_ds_read_tr16_b64_v4i16((LDS s16x4*)(smem + so + aw[c] + (32 * ks) * 256));
          s16x4 hi = __builtin_amdgcn_ds_read_tr16_b64_v4i16((LDS s16x4*)(smem + so + aw[c] + (32 * ks + 4) * 256));
          W[c][ks] = __builtin_shufflevector(lo, hi, 0, 1, 2, 3, 4, 5, 6, 7);
        }
      }
    };
    auto mm = [&](s16x8(&W)[4][2], s16x8(&X)[4][2]) {
#pragma unroll
      for (int ks = 0; ks < 2; ++ks)
#pragma unroll
        for (int c = 0; c < 4; ++c)
#pragma unroll
          for (int jj = 0; jj < 4; ++jj) {
            const int j = (c & 1) ? 3 - jj : jj;
            acc[c][j] = __builtin_amdgcn_mfma_f32_16x16x32_bf16(__builtin_bit_cast(bf16x8, W[c][ks]), __builtin_bit_cast(bf16x8, X[j][ks]), acc[c][j], 0, 0, 0);
          }
    };
    auto sched = [&]() {
#pragma unroll
      for (int i = 0; i < 32; ++i) {
        __builtin_amdgcn_sched_group_barrier(0x8, 1, 0);
        if (i < 24) __builtin_amdgcn_sched_group_barrier(0x100, 1, 0);
      }
    };
    auto rdh = [&](s16x8(&W)[4][2], s16x8(&X)[4][2], int so) {
#pragma unroll
      for (int ks = 0; ks < 2; ++ks) {
#pragma unroll
        for (int j = 0; j < 2; ++j) X[j][ks] = *(LDS s16x8*)(smem + so + ax[ks][0] + j * 2048);
#pragma unroll
        for (int c = 0; c < 4; ++c) {
          s16x4 lo = __builtin_amdgcn_ds_read_tr16_b64_v4i16((LDS s16x4*)(smem + so + aw[c] + (32 * ks) * 256));
          s16x4 hi = __builtin_amdgcn_ds_read_tr16_b64_v4i16((LDS s16x4*)(smem + so + aw[c] + (32 * ks + 4) * 256));
          W[c][ks] = __builtin_shufflevector(lo, hi, 0, 1, 2, 3, 4, 5, 6, 7);
        }
      }
    };
    auto mmj = [&](s16x8(&W)[4][2], s16x8(&X)[4][2], int j0) {
#pragma unroll
      for (int ks = 0; ks < 2; ++ks)
#pragma unroll
        for (int c = 0; c < 4; ++c)
#pragma unroll
          for (int j = j0; j < j0 + 2; ++j)
            acc[c][j] = __builtin_amdgcn_mfma_f32_16x16x32_bf16(__builtin_bit_cast(bf16x8, W[c][ks]), __builtin_bit_cast(bf16x8, X[j][ks]), acc[c][j], 0, 0, 0);
    };
    auto schedh = [&]() {
#pragma unroll
      for (int i = 0; i < 16; ++i) {
        __builtin_amdgcn_sched_group_barrier(0x8, 1, 0);
        if (i < 10) __builtin_amdgcn_sched_group_barrier(0x100, 2, 0);
      }
    };
    auto rs = __builtin_amdgcn_make_buffer_rsrc((void*)(myws + WS_SLAB), 0, NWG * 32768, 0x00020000);
    const int so_out = (partner * 4 + wid) * 8192, so_in = (b * 4 + wid) * 8192;
    u32 tag;
    constexpr int NX = 4;
    u32 in_sc;
    auto send_partial = [&]() {
      call = __builtin_amdgcn_readfirstlane(callv) + 1;
      tag = call & 1;
      float pm = 0.f;
#pragma unroll
      for (int c = 0; c < 4; ++c)
#pragma unroll
        for (int j = 2; j < 4; ++j)
#pragma unroll
          for (int r = 0; r < 4; ++r) pm = fmaxf(pm, fabsf(acc[c][j][r]));
      u32 eb = __float_as_uint(pm) & 0x7f800000u;
      eb = min(max(eb, 15u << 23), 254u << 23);
      const float up = __uint_as_float((268u << 23) - eb);
      const u32 dn = eb - (14u << 23);
#pragma unroll
      for (int c = 0; c < 4; ++c) {
        u32x4 e;
#pragma unroll
        for (int j = 0; j < 2; ++j)
#pragma unroll
          for (int r = 0; r < 2; ++r) {
            f16x2 h = __builtin_convertvector(f32x2{acc[c][2 + j][2 * r] * up, acc[c][2 + j][2 * r + 1] * up}, f16x2);
            e[j * 2 + r] = (__builtin_bit_cast(u32, h) & ~1u) | tag;
          }
        st128<AUX_SC1>(e, rs, lane * 16 + c * 1024 + so_out);
      }
      __builtin_amdgcn_raw_buffer_store_b32(dn | tag, rs, lane * 4 + 4096, so_out, AUX_SC1);
      STAMP(2);
    };
    u32x4 in[NX];
    auto load_partial = [&]() {
#pragma unroll
      for (int i = 0; i < NX; ++i) in[i] = __builtin_amdgcn_raw_buffer_load_b128(rs, lane * 16 + i * 1024, so_in, AUX_SC1);
      in_sc = __builtin_amdgcn_raw_buffer_load_b32(rs, lane * 4 + 4096, so_in, AUX_SC1);
    };
    int slot = 0;
    auto next = [&]() { slot += STAGE; if (slot == NS * STAGE) slot = 0; };
    bar();
    rd(W0, X0, slot); next();
    for (int s = 1; s < TF - 1; s += 2) {
      bar_lgkm();
      rd(W1, X1, slot); mm(W0, X0); sched(); next();
      bar_lgkm();
      rd(W0, X0, slot); mm(W1, X1); sched(); next();
    }
    bar_lgkm();
    rd(W1, X1, slot); mm(W0, X0); sched(); next();
    if (TD == 0) {
      mmj(W1, X1, 2); send_partial(); mmj(W1, X1, 0);
      load_partial();
    } else {
      bar_lgkm();
      rdh(W0, X0, slot); mmj(W1, X1, 2); schedh(); next();
      send_partial();
      mmj(W1, X1, 0);
#pragma unroll 1
      for (int h = 1; h < 2 * TD - 3; h += 2) {
        bar_lgkm();
        rdh(W1, X1, slot); mmj(W0, X0, 0); schedh(); next();
        bar_lgkm();
        rdh(W0, X0, slot); mmj(W1, X1, 0); schedh(); next();
      }
      bar_lgkm();
      rdh(W1, X1, slot); mmj(W0, X0, 0); schedh(); next();
      bar_lgkm();
      load_partial();
      rdh(W0, X0, slot); mmj(W1, X1, 0); schedh(); next();
      bar_lgkm();
      rdh(W1, X1, slot); mmj(W0, X0, 0); schedh();
      mmj(W1, X1, 0);
    }
    STAMP(1);
    bool ok = true;
    {
      u64 t0 = __builtin_amdgcn_s_memrealtime();
      u32 tm_ = 0;
      for (;;) {
        u32 an = in_sc, orr = in_sc;
#pragma unroll
        for (int i = 0; i < NX; ++i) { an &= (in[i][0] & in[i][1]) & (in[i][2] & in[i][3]); orr |= (in[i][0] | in[i][1]) | (in[i][2] | in[i][3]); }
        if (__all(((tag ? an : ~orr) & 1u) != 0)) break;
        if (give_up(myws, call, tm_, t0)) { ok = false; break; }
        asm volatile("" ::: "memory");
        load_partial();
        tm_ = ld_agent32(myws + WS_TMO);
      }
    }
    STAMP(3);
    float am = 0.f;
#pragma unroll
    for (int c = 0; c < 4; ++c)
#pragma unroll
      for (int j = 0; j < 2; ++j) {
        const float dn = __uint_as_float(in_sc & ~1u);
#pragma unroll
        for (int r = 0; r < 2; ++r) {
          float a0 = acc[c][j][2 * r], a1 = acc[c][j][2 * r + 1];
          asm("v_fma_mix_f32 %0, %1, %2, %0 op_sel_hi:[1,0,0]" : "+v"(a0) : "v"(in[c][j * 2 + r]), "v"(dn));
          asm("v_fma_mix_f32 %0, %1, %2, %0 op_sel:[1,0,0] op_sel_hi:[1,0,0]" : "+v"(a1) : "v"(in[c][j * 2 + r]), "v"(dn));
          acc[c][j][2 * r] = a0; acc[c][j][2 * r + 1] = a1;
        }
#pragma unroll
        for (int r = 0; r < 4; ++r) am = fmaxf(am, fabsf(acc[c][j][r]));
      }
    am = wave_max(am);
    if (lane == 0 && ok) {
      const u64 gr = ((u64)call << 32) | __float_as_uint(am);
      st_agent64(myws + WS_L1G + (size_t)((call & 1) * 8 + grp) * 1024 + (idx * 4 + wid) * 8, gr);
    }
#pragma unroll
    for (int c = 0; c < 4; ++c)
#pragma unroll
      for (int j = 0; j < 2; ++j) {
        int row = wm * 32 + j * 16 + (lane & 15), col = wn * 64 + c * 16 + 4 * g;
        *(LDS f32x4*)(smem + L_TILE + row * TROW + col * 4) = acc[c][j];
      }
    STAMP(4);
  }
  bar_lgkm();
  const int par = call & 1;
  f32x4 yv[4];
#pragma unroll
  for (int i = 0; i < 4; ++i) yv[i] = *(LDS f32x4*)(smem + L_TILE + rrow * TROW + (lane & 7) * 64 + i * 16);
  auto r_stg_in = __builtin_amdgcn_make_buffer_rsrc((void*)(myws + WS_STG + par * STG_BYTES), 0, (int)STG_BYTES, 0x00020000);
  const int stg_in_off = ((grp * 32 + idx) * 8 + wid) * 2048 + lane * 16;
  const int stg_probe = ((grp * 32 + idx) * 8 + wid) * 2048 + 2044;
  bool sent = false, got = false, rearmed = false;
  u32x4 e0, e1;
  u64 t0 = __builtin_amdgcn_s_memrealtime();
  const bool can_early = EARLY && dest != rank;
  auto send_early = [&]() {
    auto r_stg = __builtin_amdgcn_make_buffer_rsrc((void*)(ws_dest + WS_STG + par * STG_BYTES), 0, (int)STG_BYTES, 0x00020000);
    const int so = ((rank * 32 + ((tn & 3) + 4 * tm + 16 * kh)) * 8 + wid) * 2048;
    u32x4 a = {enc(pack_bf16(yv[0][0], yv[0][1])), enc(pack_bf16(yv[0][2], yv[0][3])), enc(pack_bf16(yv[1][0], yv[1][1])), enc(pack_bf16(yv[1][2], yv[1][3]))};
    u32x4 c = {enc(pack_bf16(yv[2][0], yv[2][1])), enc(pack_bf16(yv[2][2], yv[2][3])), enc(pack_bf16(yv[3][0], yv[3][1])), enc(pack_bf16(yv[3][2], yv[3][3]))};
    st128<AUX_SYS>(a, r_stg, lane * 16 + so);
    st128<AUX_SYS>(c, r_stg, lane * 16 + 1024 + so);
    sent = true;
    STAMP(9);
  };
  if (wid == 4) {
    const char* gsrc = myws + WS_GG + (size_t)((par * REP + (idx & (RUSE - 1))) * 64 + lane) * 8;
    u64 v; bool ok = true;
    for (;;) {
      v = ld_sys64(gsrc);
      u32 tm_ = ld_agent32(myws + WS_TMO);
      if (__all((u32)(v >> 32) == call)) { ok = tm_ != call; break; }
      if (give_up(myws, call, tm_, t0)) { ok = false; break; }
    }
    float gm = wave_max(__uint_as_float((u32)v));
    if (lane == 0) {
      lds_st(smem + L_SCALE, __float_as_uint(gm > 0.f ? 127.0f / gm : 0.f));
      __builtin_amdgcn_s_waitcnt(0xC07F);
      lds_st(smem + L_STATE, ok ? 1u : 2u);
      if (b == 0) st_agent32(myws + WS_CALL, call);
    }
    STAMP(6);
  } else {
    if (wid == 5 && idx == 0) {
      const char* l1 = myws + WS_L1G + (size_t)(par * 8 + grp) * 1024 + lane * 16;
      char* dst = ws_lane_all + WS_GG + (size_t)((par * REP + (lane >> 3)) * 64 + rank * 8 + grp) * 8;
      auto rl = __builtin_amdgcn_make_buffer_rsrc((void*)myws, 0, 1 << 20, 0x00020000);
      for (;;) {
        asm volatile("" ::: "memory");
        u32x4 v = __builtin_amdgcn_raw_buffer_load_b128(rl, (int)(l1 - myws), 0, AUX_SC1);
        u32 tm_ = ld_agent32(myws + WS_TMO);
        if (__all(v[1] == call && v[3] == call)) {
          float gm = wave_max(fmaxf(__uint_as_float(v[0]), __uint_as_float(v[2])));
          if ((lane >> 3) < RUSE) st_sys64(dst, ((u64)call << 32) | __float_as_uint(gm));
          break;
        }
        if (give_up(myws, call, tm_, t0)) break;
      }
      STAMP(5);
    }
    const u64 due = t0 + (u64)(wid - (wid > 4)) * PACE;
    while (__builtin_amdgcn_readfirstlane(lds_ld(smem + L_STATE)) == 0) {
      if (can_early && !sent && __builtin_amdgcn_s_memrealtime() >= due) {
        send_early();
      } else if (EARLY && recv && !got && (sent || !can_early)) {
        asm volatile("" ::: "memory");
        if (__builtin_amdgcn_readfirstlane(__builtin_amdgcn_raw_buffer_load_b32(r_stg_in, stg_probe, 0, AUX_SYS)) != 0) {
          e0 = __builtin_amdgcn_raw_buffer_load_b128(r_stg_in, stg_in_off, 0, AUX_SYS);
          e1 = __builtin_amdgcn_raw_buffer_load_b128(r_stg_in, stg_in_off + 1024, 0, AUX_SYS);
          got = __all(nz4(e0) && nz4(e1));
          if (got) {
            rearmed = true;
            st128<AUX_SYS>(u32x4{0, 0, 0, 0}, r_stg_in, stg_in_off);
            st128<AUX_SYS>(u32x4{0, 0, 0, 0}, r_stg_in, stg_in_off + 1024);
          }
        } else __builtin_amdgcn_s_sleep(PRE_SLEEP);
      } else {
        __builtin_amdgcn_s_sleep(1);
      }
    }
  }
  __builtin_amdgcn_s_waitcnt(0xC07F);
  const u32 state = __builtin_amdgcn_readfirstlane(lds_ld(smem + L_STATE));
  const float inv = __uint_as_float(lds_ld(smem + L_SCALE));
  if (state == 1) {
    if (!sent) {
      u32x4 o = {q4(yv[0][0], yv[0][1], yv[0][2], yv[0][3], inv), q4(yv[1][0], yv[1][1], yv[1][2], yv[1][3], inv),
                 q4(yv[2][0], yv[2][1], yv[2][2], yv[2][3], inv), q4(yv[3][0], yv[3][1], yv[3][2], yv[3][3], inv)};
      const int grow = rank * MS + m0 + (rrow >> 5) * 64 + kh * 32 + (rrow & 31);
      auto ro = __builtin_amdgcn_make_buffer_rsrc((void*)out_dest, 0, N * NSH, 0x00020000);
      st128<AUX_SYS>(o, ro, grow * NSH + (tn & 3) * BN + (lane & 7) * 16);
    }
    STAMP(7);
    if (recv) {
      while (!got) {
        asm volatile("" ::: "memory");
        u32x4 o = __builtin_amdgcn_raw_buffer_load_b128(r_myout, in_off, 0, AUX_SYS);
        u32 ps = __builtin_amdgcn_raw_buffer_load_b32(r_stg_in, stg_probe, 0, AUX_SYS);
        u32 tm_ = ld_agent32(myws + WS_TMO);
        if (__all(!(has80(o[0]) || has80(o[1]) || has80(o[2]) || has80(o[3])))) break;
        if (__builtin_amdgcn_readfirstlane(ps) != 0) {
          e0 = __builtin_amdgcn_raw_buffer_load_b128(r_stg_in, stg_in_off, 0, AUX_SYS);
          e1 = __builtin_amdgcn_raw_buffer_load_b128(r_stg_in, stg_in_off + 1024, 0, AUX_SYS);
          got = __all(nz4(e0) && nz4(e1));
        }
        if (!got && give_up(myws, call, tm_, t0)) break;
      }
      if (EARLY && got) {
        e0 -= 1u; e1 -= 1u;
        auto lo = [](u32 v) { return __uint_as_float(v << 16); };
        auto hi = [](u32 v) { return __uint_as_float(v & 0xffff0000u); };
        u32x4 o = {q4(lo(e0[0]), hi(e0[0]), lo(e0[1]), hi(e0[1]), inv), q4(lo(e0[2]), hi(e0[2]), lo(e0[3]), hi(e0[3]), inv),
                   q4(lo(e1[0]), hi(e1[0]), lo(e1[1]), hi(e1[1]), inv), q4(lo(e1[2]), hi(e1[2]), lo(e1[3]), hi(e1[3]), inv)};
        st128<AUX_SYS>(o, r_myout, in_off);
        if (!rearmed) {
          st128<AUX_SYS>(u32x4{0, 0, 0, 0}, r_stg_in, stg_in_off);
          st128<AUX_SYS>(u32x4{0, 0, 0, 0}, r_stg_in, stg_in_off + 1024);
        }
      }
      STAMP(8);
    }
  } else if (dest == rank) {
    const int grow = rank * MS + m0 + (rrow >> 5) * 64 + kh * 32 + (rrow & 31);
    st128<AUX_SYS>(u32x4{0x80808080u, 0x80808080u, 0x80808080u, 0x80808080u}, r_myout, grow * NSH + (tn & 3) * BN + (lane & 7) * 16);
  }
#endif
}

extern "C" void kernel_launch(int rank, int nranks, ncclComm_t comm, hipStream_t stream, void* const* d_in, const int* in_sizes, int n_in,
                              void* d_out, int out_size, const int* in_dtypes, int out_dtype, void* const* d_out_all,
                              void* const* d_ws_all, size_t ws_bytes) {
  if (nranks != NR || (unsigned)rank >= (unsigned)NR || ws_bytes < WS_NEED || n_in < 2 || in_sizes[0] != MS * K || in_sizes[1] != K * N || out_size != N * NSH) {
    fprintf(stderr, "unexpected problem shape\n");
    return;
  }
  Params P;
  P.x = (const u16*)d_in[0];
  P.w = (const u16*)d_in[1];
  for (int p = 0; p < NR; ++p) { P.out[p] = (char*)d_out_all[p]; P.ws[p] = (char*)d_ws_all[p]; }
  P.myws = P.ws[rank]; P.myout = (char*)d_out;
  P.rank = rank;
  hipError_t e = hipFuncSetAttribute((const void*)fused_gemm_a2a_q8, hipFuncAttributeMaxDynamicSharedMemorySize, LDS_BYTES);
  if (e != hipSuccess) { fprintf(stderr, "hipFuncSetAttribute: %s\n", hipGetErrorString(e)); return; }
  hipLaunchKernelGGL(fused_gemm_a2a_q8, dim3(NWG), dim3(512), LDS_BYTES, stream, P);
}
```
